```python
import numpy as np

kernel_name = "gpu_gemm_rs_m4096_k4096_n16384_f32_relu"


ABSMAX_RTOL = 0.001

COMPUTE_DTYPE = "float32"
EPILOGUE = "relu"
IN_QUANT = None
OUT_QUANT = None
IN_DTYPES = {"x": "float32", "w_mat": "float32"}
WORLD_SIZE = 8

def setup_inputs():

    x = np.random.randn(4096, 4096).astype(np.float32)
    w_mat = (np.random.randn(4096, 16384) / np.sqrt(4096)).astype(np.float32)
    return {"x": x, "w_mat": w_mat}

def _epilogue(y):
    return np.maximum(y, 0.0).astype(np.float32)

def reference(x, w_mat):
    y = (x @ w_mat).astype(np.float32)
    return _epilogue(y)
```

```cpp
#include <hip/hip_runtime.h>
#include <rccl/rccl.h>
#include <cstdio>
#include <cstdint>

#define HIP_CHECK(expr) do { hipError_t _e = (expr); if (_e != hipSuccess) { \
    fprintf(stderr, "HIP error %s at %s:%d\n", hipGetErrorString(_e), __FILE__, __LINE__); \
    return; } } while (0)

namespace {

typedef _Float16 h8 __attribute__((ext_vector_type(8)));
typedef float f4 __attribute__((ext_vector_type(4)));
typedef unsigned u4 __attribute__((ext_vector_type(4)));

constexpr int NRANK = 8;
constexpr int NCOL = 16384;
constexpr int KS = 512;
constexpr int BN = 128;
constexpr int NCT = NCOL / BN;
constexpr int NWG = 256;
constexpr int NGEMM = 224;
constexpr int NTHR = 512;
constexpr int PWAVES = 32;

constexpr size_t KiB = 1024, MiB = 1024 * 1024;
constexpr unsigned OFF_TICKET = 0;
constexpr unsigned OFF_TMO = 128;
constexpr unsigned OFF_CX = 256;
constexpr unsigned OFF_CW = 384;
constexpr unsigned OFF_XRDY = 1024;
constexpr unsigned OFF_WRDY = 2048;
constexpr unsigned OFF_XCNT = 8192;
constexpr unsigned OFF_WCNT = 16384;
constexpr unsigned OFF_FLC = 65536;
constexpr unsigned OFF_FLO = 69632;
constexpr unsigned OFF_DBG = 131072;
constexpr size_t OFF_WC = 1 * MiB;
constexpr size_t OFF_XC = OFF_WC + 32 * MiB;
constexpr size_t OFF_WP = OFF_XC + 8 * MiB;
constexpr size_t OFF_XP = OFF_WP + 48 * MiB;
constexpr size_t OFF_SO = OFF_XP + 32 * MiB;
constexpr size_t OFF_SP = OFF_SO + 32 * MiB;
constexpr size_t WS_NEED = OFF_SP + 64 * MiB;

constexpr unsigned STAGE_B = 49152;
constexpr unsigned LDS_BC = 3 * STAGE_B;
constexpr unsigned LDS_BYTES = LDS_BC + 64;

constexpr unsigned long long DEADLINE_TICKS = 50000000ull;

__device__ __forceinline__ int plane_elem(int v, int i) {
  return (int)((0x6afe3ea68f8ec3a2ull >> ((v - 1) * 9 + i * 3)) & 7ull);
}
__device__ __forceinline__ int dir_of_f(int f) { return (0x73645210u >> (4 * f)) & 7; }
__device__ __forceinline__ int f_of_dir(int e) { return (0x75346210u >> (4 * e)) & 7; }

struct Params {
  const float* x;
  const float* w;
  float* out;
  char* wsx[NRANK];
  int rank;
};
__device__ __forceinline__ char* peer_ws(const Params& P, int e) {
  char* p = P.wsx[1];
  p = (e == 2) ? P.wsx[2] : p; p = (e == 3) ? P.wsx[3] : p; p = (e == 4) ? P.wsx[4] : p;
  p = (e == 5) ? P.wsx[5] : p; p = (e == 6) ? P.wsx[6] : p; p = (e == 7) ? P.wsx[7] : p;
  return p;
}

typedef __attribute__((address_space(1))) unsigned gu32;

__device__ __forceinline__ unsigned ld_agent(const char* p) {
  return __hip_atomic_load((const unsigned*)p, __ATOMIC_RELAXED, __HIP_MEMORY_SCOPE_AGENT);
}
__device__ __forceinline__ unsigned ld_sys(const char* p) {
  return __hip_atomic_load((const unsigned*)p, __ATOMIC_RELAXED, __HIP_MEMORY_SCOPE_SYSTEM);
}
__device__ __forceinline__ void st_sys(char* p, unsigned v) {
  __hip_atomic_store((unsigned*)p, v, __ATOMIC_RELAXED, __HIP_MEMORY_SCOPE_SYSTEM);
}
__device__ __forceinline__ void add_agent(char* p, unsigned v) {
  (void)__hip_atomic_fetch_add((unsigned*)p, v, __ATOMIC_RELAXED, __HIP_MEMORY_SCOPE_AGENT);
}
#define VMCNT0() asm volatile("s_waitcnt vmcnt(0)" ::: "memory")

__device__ __forceinline__ bool wave_wait(const char* word, unsigned target, char* tmo, unsigned call) {
  unsigned long long t0 = __builtin_amdgcn_s_memrealtime();
  for (;;) {
    unsigned v = ld_sys(word);
    if (__all((int)(v - target) >= 0)) return true;
    if (ld_agent(tmo) == call || __builtin_amdgcn_s_memrealtime() - t0 > DEADLINE_TICKS) {
      __hip_atomic_store((unsigned*)tmo, call, __ATOMIC_RELAXED, __HIP_MEMORY_SCOPE_AGENT);
      return false;
    }
    __builtin_amdgcn_s_sleep(2);
  }
}

__device__ __forceinline__ unsigned pack2(float a, float b) {
  typedef _Float16 h2 __attribute__((ext_vector_type(2)));
  h2 v = {(_Float16)a, (_Float16)b};
  return __builtin_bit_cast(unsigned, v);
}

__device__ __forceinline__ void gemm_issue(char* smem, int t, int buf, const char* a0, const char* a1, const char* a2, const char* a3,
                                           const char* b0, const char* b1, const char* b2, const char* b3, int wave, int lane) {
  int sh = t >> 3, kk = t & 7;
  const char* a = (sh == 0) ? a0 : (sh == 1) ? a1 : (sh == 2) ? a2 : a3;
  const char* b = (sh == 0) ? b0 : (sh == 1) ? b1 : (sh == 2) ? b2 : b3;
  a += kk * 32768 + wave * 4096 + lane * 16;
  b += kk * 16384 + wave * 2048 + lane * 16;
  char* dst = smem + buf * STAGE_B;
  #pragma unroll
  for (int i = 0; i < 4; ++i)
    __builtin_amdgcn_global_load_lds((const unsigned*)(a + i * 1024), (unsigned*)(dst + wave * 4096 + i * 1024), 16, 0, 0);
  #pragma unroll
  for (int i = 0; i < 2; ++i)
    __builtin_amdgcn_global_load_lds((const unsigned*)(b + i * 1024), (unsigned*)(dst + 32768 + wave * 2048 + i * 1024), 16, 0, 0);
}

#define COPY2(soff, doff) do { \
      u4 v0_ = __builtin_amdgcn_raw_buffer_load_b128(Rr, (soff), 0, 17); \
      u4 v1_ = __builtin_amdgcn_raw_buffer_load_b128(Rr, (soff) + 1024, 0, 17); \
      VMCNT0(); \
      if (pending) { if (lane == 0) add_agent(lws + pending, 1u); pending = 0; } \
      __builtin_amdgcn_raw_buffer_store_b128(v0_, Lr, (doff), 0, 17); \
      __builtin_amdgcn_raw_buffer_store_b128(v1_, Lr, (doff) + 1024, 0, 17); \
    } while (0)
#define FLUSH() do { VMCNT0(); if (pending) { if (lane == 0) add_agent(lws + pending, 1u); pending = 0; } } while (0)

#define STAMP(slot) do {} while (0)

__global__ void __launch_bounds__(NTHR, 2) fused_gemm_rs(Params P) {
  extern __shared__ __attribute__((aligned(16))) char smem[];
  const int tid = threadIdx.x, lane = tid & 63;
  const int wave = __builtin_amdgcn_readfirstlane(tid >> 6);
  const int bid = blockIdx.x;
  const int rank = P.rank;
  char* const lws = P.wsx[0];
  char* const tmo = lws + OFF_TMO;

  STAMP(0);
  if (tid == 0) {
    unsigned t = __hip_atomic_fetch_add((unsigned*)(lws + OFF_TICKET), 1u, __ATOMIC_RELAXED, __HIP_MEMORY_SCOPE_AGENT);
    *(volatile unsigned*)(smem + LDS_BC) = t / NWG + 1;
  }
  f4 xv[2][2];
  {
    #pragma unroll
    for (int i = 0; i < 2; ++i) {
      int F = (bid * 8 + wave) * 2 + i;
      int rh = F >> 8, kb = (F >> 4) & 15, rb = F & 15;
      const float* src = P.x + (size_t)(rh * 256 + rb * 16 + (lane & 15)) * KS + kb * 32 + (lane >> 4) * 8;
      xv[i][0] = *(const f4*)src;
      xv[i][1] = *(const f4*)(src + 4);
    }
  }
  __syncthreads();
  const unsigned call = __builtin_amdgcn_readfirstlane(*(volatile unsigned*)(smem + LDS_BC));
  const unsigned par = call & 1;

  auto Lr = __builtin_amdgcn_make_buffer_rsrc((void*)lws, 0, (int)WS_NEED, 0x00020000);

  {
    #pragma unroll
    for (int i = 0; i < 2; ++i) {
      int F = (bid * 8 + wave) * 2 + i;
      u4 o;
      o[0] = pack2(xv[i][0][0], xv[i][0][1]);
      o[1] = pack2(xv[i][0][2], xv[i][0][3]);
      o[2] = pack2(xv[i][1][0], xv[i][1][1]);
      o[3] = pack2(xv[i][1][2], xv[i][1][3]);
      __builtin_amdgcn_raw_buffer_store_b128(o, Lr, (int)(OFF_XC + par * 4 * MiB) + F * 1024 + lane * 16, 0, 17);
    }
    VMCNT0();
    __syncthreads();
    if (tid == 0) {
      unsigned old = __hip_atomic_fetch_add((unsigned*)(lws + OFF_CX), 1u, __ATOMIC_RELAXED, __HIP_MEMORY_SCOPE_AGENT);
      if (old == call * NWG - 1) {
        __builtin_amdgcn_fence(__ATOMIC_RELEASE, "");
        VMCNT0();
        #pragma unroll
        for (int e = 1; e < 8; ++e) st_sys(P.wsx[e] + OFF_XRDY + e * 128, call);
      }
    }
  }
  STAMP(1);

  if (bid < NGEMM) {
    {
      const int q = lane & 3, kq = (lane >> 2) & 3, nb = lane >> 4;
      for (int T = bid * 8 + wave; T < NCT * 16 * 2; T += NGEMM * 8) {
        int nh = T & 1, kb = (T >> 1) & 15, c = T >> 5;
        const float* src = P.w + (size_t)(kb * 32 + kq * 8) * NCOL + c * BN + nh * 64 + nb * 16 + q * 4;
        f4 v[8];
        #pragma unroll
        for (int j = 0; j < 8; ++j) v[j] = __builtin_nontemporal_load((const f4*)(src + (size_t)j * NCOL));
        int dst = (int)(OFF_WC + par * 16 * MiB) + c * 131072 + kb * 8192 + (nh * 4 + nb) * 1024 + (kq * 16 + q * 4) * 16;
        #pragma unroll
        for (int i = 0; i < 4; ++i) {
          u4 o;
          o[0] = pack2(v[0][i], v[1][i]);
          o[1] = pack2(v[2][i], v[3][i]);
          o[2] = pack2(v[4][i], v[5][i]);
          o[3] = pack2(v[6][i], v[7][i]);
          __builtin_amdgcn_raw_buffer_store_b128(o, Lr, dst + i * 16, 0, 17);
        }
      }
      VMCNT0();
      __syncthreads();
      if (tid == 0) {
        unsigned old = __hip_atomic_fetch_add((unsigned*)(lws + OFF_CW), 1u, __ATOMIC_RELAXED, __HIP_MEMORY_SCOPE_AGENT);
        if (old == call * NGEMM - 1) {
          __builtin_amdgcn_fence(__ATOMIC_RELEASE, "");
          VMCNT0();
          #pragma unroll
          for (int e = 1; e < 8; ++e) st_sys(P.wsx[e] + OFF_WRDY + e * 128, call);
        }
      }
    }
    STAMP(2);

    const int xcd = bid & 7, slot = bid >> 3;
    const int wm = wave >> 1, wn = wave & 1;
    int tcount = 0;
    for (int u = slot; u < 64; u += NGEMM / 8, ++tcount) {
      const int c = xcd + 8 * (u >> 2), rb = u & 3;
      const int j = c % 7, f = j + 1;
      const int dj = dir_of_f(f);
      const int dq = (rb < 2) ? dj : 0;
      const int h = rb & 1;
      const int e0 = plane_elem(f, 0), e1 = plane_elem(f, 1), e2 = plane_elem(f, 2);
      if (wave == 0) {
        const char* word; unsigned target;
        int l = lane & 7;
        if (l < 3) { word = lws + OFF_WCNT + (c * 3 + l) * 128; target = call * PWAVES; }
        else if (l < 6) { int e = (l == 3) ? e0 : (l == 4) ? e1 : e2; word = lws + OFF_XCNT + (e * 8 + dq) * 128; target = call * PWAVES; }
        else if (l == 6) { word = lws + OFF_CX; target = call * NWG; }
        else { word = lws + OFF_CW; target = call * NGEMM; }
        (void)wave_wait(word, target, tmo, call);
        __builtin_amdgcn_fence(__ATOMIC_ACQUIRE, "agent");
        VMCNT0();
      }
      __syncthreads();
      STAMP(3 + tcount * 3);

      const char* a0 = lws + OFF_XC + par * 4 * MiB + (size_t)(((rank ^ dq) * 2 + h)) * 262144;
      const char* a1 = lws + OFF_XP + (size_t)(e0 * 8 + dq) * 524288 + h * 262144;
      const char* a2 = lws + OFF_XP + (size_t)(e1 * 8 + dq) * 524288 + h * 262144;
      const char* a3 = lws + OFF_XP + (size_t)(e2 * 8 + dq) * 524288 + h * 262144;
      const char* b0 = lws + OFF_WC + par * 16 * MiB + (size_t)c * 131072;
      const char* b1 = lws + OFF_WP + (size_t)(c * 3 + 0) * 131072;
      const char* b2 = lws + OFF_WP + (size_t)(c * 3 + 1) * 131072;
      const char* b3 = lws + OFF_WP + (size_t)(c * 3 + 2) * 131072;

      f4 acc[4][4];
      #pragma unroll
      for (int mi = 0; mi < 4; ++mi)
        #pragma unroll
        for (int ni = 0; ni < 4; ++ni) acc[mi][ni] = f4{0.f, 0.f, 0.f, 0.f};

      gemm_issue(smem, 0, 0, a0, a1, a2, a3, b0, b1, b2, b3, wave, lane);
      gemm_issue(smem, 1, 1, a0, a1, a2, a3, b0, b1, b2, b3, wave, lane);
      int buf = 0;
      for (int t = 0; t < 32; ++t) {
        if (t < 31) asm volatile("s_waitcnt vmcnt(6)" ::: "memory");
        else asm volatile("s_waitcnt vmcnt(0)" ::: "memory");
        asm volatile("s_waitcnt lgkmcnt(0)" ::: "memory");
        __builtin_amdgcn_s_barrier();
        int nbuf = buf + 2; if (nbuf >= 3) nbuf -= 3;
        if (t + 2 < 32) gemm_issue(smem, t + 2, nbuf, a0, a1, a2, a3, b0, b1, b2, b3, wave, lane);
        const char* sa = smem + buf * STAGE_B + wm * 4096 + lane * 16;
        const char* sb = smem + buf * STAGE_B + 32768 + wn * 4096 + lane * 16;
        #pragma unroll
        for (int kb = 0; kb < 2; ++kb) {
          h8 af[4], bf[4];
          #pragma unroll
          for (int i = 0; i < 4; ++i) af[i] = *(const h8*)(sa + kb * 16384 + i * 1024);
          #pragma unroll
          for (int i = 0; i < 4; ++i) bf[i] = *(const h8*)(sb + kb * 8192 + i * 1024);
          #pragma unroll
          for (int mi = 0; mi < 4; ++mi)
            #pragma unroll
            for (int ni = 0; ni < 4; ++ni)
              acc[mi][ni] = __builtin_amdgcn_mfma_f32_16x16x32_f16(bf[ni], af[mi], acc[mi][ni], 0, 0, 0);
        }
        buf = buf + 1; if (buf >= 3) buf = 0;
      }
      STAMP(4 + tcount * 3);
      {
        int dst = (rb < 2) ? (int)(OFF_SP + par * 32 * MiB) : (int)OFF_SO;
        dst += (c * 2 + h) * 131072 + wave * 16384 + lane * 16;
        #pragma unroll
        for (int mi = 0; mi < 4; ++mi)
          #pragma unroll
          for (int ni = 0; ni < 4; ++ni)
            __builtin_amdgcn_raw_buffer_store_b128(__builtin_bit_cast(u4, acc[mi][ni]), Lr, dst + (mi * 4 + ni) * 1024, 0, 17);
        VMCNT0();
        __syncthreads();
        if (tid == 0) {
          int fu = (j * 64 + (c / 7) * 2 + h) * 4;
          if (rb < 2) st_sys(peer_ws(P, dj) + OFF_FLC + fu, call);
          else st_sys(lws + OFF_FLO + fu, call);
        }
      }
      STAMP(5 + tcount * 3);
    }
  } else if (bid < NGEMM + 28) {
    const int pw = bid - NGEMM;
    const int e = (pw % 7) + 1, sub = pw / 7;
    const int wv = sub * 8 + wave;
    char* const rws = peer_ws(P, e);
    auto Rr = __builtin_amdgcn_make_buffer_rsrc((void*)rws, 0, (int)WS_NEED, 0x00020000);
    unsigned pending = 0;

    bool ok = wave_wait(lws + OFF_XRDY + e * 128, call, tmo, call);
    STAMP(2);
    if (ok) {
      for (int b = 0; b < 4; ++b) {
        int dq = (b == 0) ? 0 : dir_of_f(plane_elem(e, b - 1));
        int soff = (int)(OFF_XC + par * 4 * MiB) + (rank ^ dq) * 524288 + wv * 2048 + lane * 16;
        int doff = (int)OFF_XP + (e * 8 + dq) * 524288 + wv * 2048 + lane * 16;
        for (int it = 0; it < 8; ++it) COPY2(soff + it * 65536, doff + it * 65536);
        pending = OFF_XCNT + (e * 8 + dq) * 128;
      }
      FLUSH();
    }
    STAMP(3);
    ok = ok && wave_wait(lws + OFF_WRDY + e * 128, call, tmo, call);
    if (ok) {
      for (int c = 0; c < NCT; ++c) {
        int f = c % 7 + 1;
        if (__builtin_popcount(f & e) & 1) continue;
        int i = (plane_elem(f, 0) == e) ? 0 : (plane_elem(f, 1) == e) ? 1 : 2;
        int soff = (int)(OFF_WC + par * 16 * MiB) + c * 131072 + wv * 2048 + lane * 16;
        int doff = (int)OFF_WP + (c * 3 + i) * 131072 + wv * 2048 + lane * 16;
        COPY2(soff, doff);
        COPY2(soff + 65536, doff + 65536);
        pending = OFF_WCNT + (c * 3 + i) * 128;
      }
      FLUSH();
    }
    STAMP(4);
    {
      const int j = f_of_dir(e) - 1;
      const int ntile = ((NCT - j + 6) / 7) * 2;
      const int wt = wv >> 2, mi = wv & 3;
      int ready = 0;
      for (int u = 0; u < ntile; ++u) {
        if (ok && u >= ready) {
          unsigned long long t0 = __builtin_amdgcn_s_memrealtime();
          for (;;) {
            int uu = u + lane; if (uu > 63) uu = 63;
            unsigned a = ld_sys(lws + OFF_FLC + (j * 64 + uu) * 4);
            unsigned b = ld_sys(lws + OFF_FLO + (j * 64 + uu) * 4);
            bool good = ((int)(a - call) >= 0) && ((int)(b - call) >= 0) && (u + lane < ntile);
            unsigned long long m = __ballot(good);
            int n = (~m == 0ull) ? 64 : __builtin_ctzll(~m);
            if (n > 0) { ready = u + n; break; }
            if (ld_agent(tmo) == call || __builtin_amdgcn_s_memrealtime() - t0 > DEADLINE_TICKS) {
              __hip_atomic_store((unsigned*)tmo, call, __ATOMIC_RELAXED, __HIP_MEMORY_SCOPE_AGENT);
              ok = false; break;
            }
            __builtin_amdgcn_s_sleep(2);
          }
          asm volatile("" ::: "memory");
        }
        const int c = j + 7 * (u >> 1), h = u & 1;
        const int toff = (c * 2 + h) * 131072 + wv * 4096 + lane * 16;
        float* o = P.out + (size_t)(h * 256 + (wt >> 1) * 64 + mi * 16 + (lane & 15)) * NCOL + c * BN + (wt & 1) * 64 + (lane >> 4) * 4;
        #pragma unroll
        for (int it = 0; it < 2; ++it) {
          f4 r0, r1;
          if (ok) {
            u4 a0 = __builtin_amdgcn_raw_buffer_load_b128(Rr, (int)(OFF_SP + par * 32 * MiB) + toff + it * 2048, 0, 17);
            u4 a1 = __builtin_amdgcn_raw_buffer_load_b128(Rr, (int)(OFF_SP + par * 32 * MiB) + toff + it * 2048 + 1024, 0, 17);
            u4 b0 = __builtin_amdgcn_raw_buffer_load_b128(Lr, (int)OFF_SO + toff + it * 2048, 0, 17);
            u4 b1 = __builtin_amdgcn_raw_buffer_load_b128(Lr, (int)OFF_SO + toff + it * 2048 + 1024, 0, 17);
            r0 = __builtin_bit_cast(f4, a0) + __builtin_bit_cast(f4, b0);
            r1 = __builtin_bit_cast(f4, a1) + __builtin_bit_cast(f4, b1);
            #pragma unroll
            for (int k = 0; k < 4; ++k) { r0[k] = fmaxf(r0[k], 0.f); r1[k] = fmaxf(r1[k], 0.f); }
          } else {
            float nanv = __builtin_nanf("");
            r0 = f4{nanv, nanv, nanv, nanv}; r1 = r0;
          }
          *(f4*)(o + it * 32) = r0;
          *(f4*)(o + it * 32 + 16) = r1;
        }
      }
    }
    STAMP(5);
  }
}

}

extern "C" void kernel_launch(int rank, int nranks, ncclComm_t comm,
                              hipStream_t stream,
                              void* const* d_in, const int* in_sizes, int n_in,
                              void* d_out, int out_size,
                              const int* in_dtypes, int out_dtype,
                              void* const* d_out_all,
                              void* const* d_ws_all, size_t ws_bytes) {
  (void)comm; (void)in_sizes; (void)n_in; (void)out_size; (void)in_dtypes; (void)out_dtype; (void)d_out_all;
  if (nranks != NRANK || ws_bytes < WS_NEED) {
    fprintf(stderr, "unsupported configuration: nranks %d ws %zu\n", nranks, ws_bytes);
    return;
  }
  HIP_CHECK(hipFuncSetAttribute((const void*)fused_gemm_rs, hipFuncAttributeMaxDynamicSharedMemorySize, LDS_BYTES));
  Params P;
  P.x = (const float*)d_in[0];
  P.w = (const float*)d_in[1];
  P.out = (float*)d_out;
  for (int e = 0; e < NRANK; ++e) P.wsx[e] = (char*)d_ws_all[rank ^ e];
  P.rank = rank;
  fused_gemm_rs<<<NWG, NTHR, LDS_BYTES, stream>>>(P);
  HIP_CHECK(hipGetLastError());
}
```

```cpp
#include <hip/hip_runtime.h>
#include <rccl/rccl.h>
#include <cstdio>
#include <cstdint>

#define HIP_CHECK(expr) do { hipError_t _e = (expr); if (_e != hipSuccess) { \
    fprintf(stderr, "HIP error %s at %s:%d\n", hipGetErrorString(_e), __FILE__, __LINE__); \
    return; } } while (0)

namespace {

typedef _Float16 h8 __attribute__((ext_vector_type(8)));
typedef float f4 __attribute__((ext_vector_type(4)));
typedef unsigned u4 __attribute__((ext_vector_type(4)));

constexpr int NRANK = 8;
constexpr int NCOL = 16384;
constexpr int KS = 512;
constexpr int BN = 128;
constexpr int NCT = NCOL / BN;
constexpr int NWG = 256;
constexpr int NGEMM = 224;
constexpr int NTHR = 512;
#define PW_PER_WG 8
constexpr int PWAVES = 4 * PW_PER_WG;

constexpr size_t KiB = 1024, MiB = 1024 * 1024;
constexpr unsigned OFF_TICKET = 0;
constexpr unsigned OFF_TMO = 128;
constexpr unsigned OFF_CX = 256;
constexpr unsigned OFF_CW = 384;
constexpr unsigned OFF_XRDY = 1024;
constexpr unsigned OFF_WRDY = 2048;
constexpr unsigned OFF_Q = 3072;
constexpr unsigned OFF_XCNT = 8192;
constexpr unsigned OFF_WCNT = 16384;
constexpr unsigned OFF_FLC = 65536;
constexpr unsigned OFF_FLO = 69632;
constexpr unsigned OFF_DBG = 131072;
constexpr size_t OFF_WC = 1 * MiB;
constexpr size_t OFF_XC = OFF_WC + 32 * MiB;
constexpr size_t OFF_WP = OFF_XC + 8 * MiB;
constexpr size_t OFF_XP = OFF_WP + 48 * MiB;
constexpr size_t OFF_SO = OFF_XP + 32 * MiB;
constexpr size_t OFF_SP = OFF_SO + 32 * MiB;
constexpr size_t WS_NEED = OFF_SP + 64 * MiB;

constexpr unsigned STAGE_B = 49152;
constexpr unsigned LDS_BC = 3 * STAGE_B;
constexpr unsigned LDS_BYTES = LDS_BC + 64;

constexpr unsigned long long DEADLINE_TICKS = 50000000ull;

__device__ __forceinline__ int plane_elem(int v, int i) {
  return (int)((0x6afe3ea68f8ec3a2ull >> ((v - 1) * 9 + i * 3)) & 7ull);
}
__device__ __forceinline__ int dir_of_f(int f) { return (0x73645210u >> (4 * f)) & 7; }
__device__ __forceinline__ int f_of_dir(int e) { return (0x75346210u >> (4 * e)) & 7; }

struct Params {
  const float* x;
  const float* w;
  float* out;
  char* wsx[NRANK];
  int rank;
};
__device__ __forceinline__ char* peer_ws(const Params& P, int e) {
  char* p = P.wsx[1];
  p = (e == 2) ? P.wsx[2] : p; p = (e == 3) ? P.wsx[3] : p; p = (e == 4) ? P.wsx[4] : p;
  p = (e == 5) ? P.wsx[5] : p; p = (e == 6) ? P.wsx[6] : p; p = (e == 7) ? P.wsx[7] : p;
  return p;
}

typedef __attribute__((address_space(1))) unsigned gu32;

__device__ __forceinline__ unsigned ld_agent(const char* p) {
  return __hip_atomic_load((const unsigned*)p, __ATOMIC_RELAXED, __HIP_MEMORY_SCOPE_AGENT);
}
__device__ __forceinline__ unsigned ld_sys(const char* p) {
  return __hip_atomic_load((const unsigned*)p, __ATOMIC_RELAXED, __HIP_MEMORY_SCOPE_SYSTEM);
}
__device__ __forceinline__ void st_sys(char* p, unsigned v) {
  __hip_atomic_store((unsigned*)p, v, __ATOMIC_RELAXED, __HIP_MEMORY_SCOPE_SYSTEM);
}
__device__ __forceinline__ void add_agent(char* p, unsigned v) {
  (void)__hip_atomic_fetch_add((unsigned*)p, v, __ATOMIC_RELAXED, __HIP_MEMORY_SCOPE_AGENT);
}
#define VMCNT0() asm volatile("s_waitcnt vmcnt(0)" ::: "memory")

template <int SLEEP>
__device__ __forceinline__ bool wave_wait_s(const char* word, unsigned target, char* tmo, unsigned call) {
  unsigned long long t0 = __builtin_amdgcn_s_memrealtime();
  for (;;) {
    unsigned v = ld_sys(word);
    if (__all((int)(v - target) >= 0)) return true;
    if (ld_agent(tmo) == call || __builtin_amdgcn_s_memrealtime() - t0 > DEADLINE_TICKS) {
      __hip_atomic_store((unsigned*)tmo, call, __ATOMIC_RELAXED, __HIP_MEMORY_SCOPE_AGENT);
      return false;
    }
    __builtin_amdgcn_s_sleep(SLEEP);
  }
}
#define wave_wait wave_wait_s<2>
#define GSLEEP 64

#define wave_wait_nl wave_wait
__device__ __forceinline__ unsigned pack2(float a, float b) {
  typedef _Float16 h2 __attribute__((ext_vector_type(2)));
  h2 v = {(_Float16)a, (_Float16)b};
  return __builtin_bit_cast(unsigned, v);
}

__device__ __forceinline__ void gemm_issue(char* smem, int t, int buf, const char* a0, const char* a1, const char* a2, const char* a3,
                                           const char* b0, const char* b1, const char* b2, const char* b3, int wave, int lane) {
  int sh = t >> 3, kk = t & 7;
  const char* a = (sh == 0) ? a0 : (sh == 1) ? a1 : (sh == 2) ? a2 : a3;
  const char* b = (sh == 0) ? b0 : (sh == 1) ? b1 : (sh == 2) ? b2 : b3;
  a += kk * 32768 + wave * 4096 + lane * 16;
  b += kk * 16384 + wave * 2048 + lane * 16;
  char* dst = smem + buf * STAGE_B;
  #pragma unroll
  for (int i = 0; i < 4; ++i)
    __builtin_amdgcn_global_load_lds((const unsigned*)(a + i * 1024), (unsigned*)(dst + wave * 4096 + i * 1024), 16, 0, 0);
  #pragma unroll
  for (int i = 0; i < 2; ++i)
    __builtin_amdgcn_global_load_lds((const unsigned*)(b + i * 1024), (unsigned*)(dst + 32768 + wave * 2048 + i * 1024), 16, 0, 0);
}

#define GLOAD2(D0, D1, BASE, VOFF, IMM) \
  asm volatile("global_load_dwordx4 %0, %2, %3 offset:%4 sc0 sc1\n\tglobal_load_dwordx4 %1, %2, %3 offset:%5 sc0 sc1" \
               : "=&v"(D0), "=&v"(D1) : "v"(VOFF), "s"(BASE), "i"(IMM), "i"(IMM + 1024) : "memory")
#define GSTORE2(D0, D1, BASE, VOFF, IMM) \
  asm volatile("global_store_dwordx4 %2, %0, %3 offset:%4 sc0 sc1\n\tglobal_store_dwordx4 %2, %1, %3 offset:%5 sc0 sc1\n\ts_nop 1" \
               :: "v"(D0), "v"(D1), "v"(VOFF), "s"(BASE), "i"(IMM), "i"(IMM + 1024) : "memory")
#define DRAW() do { if (lane == 0) asm volatile("global_atomic_add %0, %1, %2, %3 sc0" : "+v"(tv) : "v"(qoff), "v"(1u), "s"(lws) : "memory"); } while (0)
#define SIGNAL(OFFS) do { if (lane == 0) asm volatile("global_atomic_add %0, %1, %2" :: "v"(OFFS), "v"(1u), "s"(lws) : "memory"); } while (0)

#define STAMP(slot) do {} while (0)

__global__ void __launch_bounds__(NTHR, 2) fused_gemm_rs(Params P) {
  extern __shared__ __attribute__((aligned(16))) char smem[];
  const int tid = threadIdx.x, lane = tid & 63;
  const int wave = __builtin_amdgcn_readfirstlane(tid >> 6);
  const int bid = blockIdx.x;
  const int rank = P.rank;
  char* const lws = P.wsx[0];
  char* const tmo = lws + OFF_TMO;

  STAMP(0);
  if (tid == 0) {
    unsigned t = __hip_atomic_fetch_add((unsigned*)(lws + OFF_TICKET), 1u, __ATOMIC_RELAXED, __HIP_MEMORY_SCOPE_AGENT);
    *(volatile unsigned*)(smem + LDS_BC) = t / NWG + 1;
  }
  f4 xv[2][2];
  {
    #pragma unroll
    for (int i = 0; i < 2; ++i) {
      int F = (bid * 8 + wave) * 2 + i;
      int rh = F >> 8, kb = (F >> 4) & 15, rb = F & 15;
      const float* src = P.x + (size_t)(rh * 256 + rb * 16 + (lane & 15)) * KS + kb * 32 + (lane >> 4) * 8;
      xv[i][0] = *(const f4*)src;
      xv[i][1] = *(const f4*)(src + 4);
    }
  }
  __syncthreads();
  const unsigned call = __builtin_amdgcn_readfirstlane(*(volatile unsigned*)(smem + LDS_BC));
  const unsigned par = call & 1;

  auto Lr = __builtin_amdgcn_make_buffer_rsrc((void*)lws, 0, (int)WS_NEED, 0x00020000);

  {
    #pragma unroll
    for (int i = 0; i < 2; ++i) {
      int F = (bid * 8 + wave) * 2 + i;
      u4 o;
      o[0] = pack2(xv[i][0][0], xv[i][0][1]);
      o[1] = pack2(xv[i][0][2], xv[i][0][3]);
      o[2] = pack2(xv[i][1][0], xv[i][1][1]);
      o[3] = pack2(xv[i][1][2], xv[i][1][3]);
      __builtin_amdgcn_raw_buffer_store_b128(o, Lr, (int)(OFF_XC + par * 4 * MiB) + F * 1024 + lane * 16, 0, 17);
    }
    VMCNT0();
    __syncthreads();
    if (tid == 0) {
      unsigned old = __hip_atomic_fetch_add((unsigned*)(lws + OFF_CX), 1u, __ATOMIC_RELAXED, __HIP_MEMORY_SCOPE_AGENT);
      if (old == call * NWG - 1) {
        __builtin_amdgcn_fence(__ATOMIC_RELEASE, "");
        VMCNT0();
        #pragma unroll
        for (int e = 1; e < 8; ++e) st_sys(P.wsx[e] + OFF_XRDY + e * 128, call);
      }
    }
  }
  STAMP(1);

  if (bid < NGEMM) {
    {
      const int kq = lane >> 4, col = lane & 15;
      for (int T = bid * 8 + wave; T < NCT * 16 * 2; T += NGEMM * 8) {
        int nh = T & 1, kb = (T >> 1) & 15, c = T >> 5;
        const float* src = P.w + (size_t)(kb * 32 + kq * 8) * NCOL + c * BN + nh * 64 + col;
        float v[4][8];
        #pragma unroll
        for (int j = 0; j < 8; ++j)
          #pragma unroll
          for (int i = 0; i < 4; ++i) v[i][j] = src[(size_t)j * NCOL + i * 16];
        int dst = (int)(OFF_WC + par * 16 * MiB) + c * 131072 + kb * 8192 + nh * 4096 + lane * 16;
        #pragma unroll
        for (int i = 0; i < 4; ++i) {
          u4 o;
          o[0] = pack2(v[i][0], v[i][1]);
          o[1] = pack2(v[i][2], v[i][3]);
          o[2] = pack2(v[i][4], v[i][5]);
          o[3] = pack2(v[i][6], v[i][7]);
          __builtin_amdgcn_raw_buffer_store_b128(o, Lr, dst + i * 1024, 0, 17);
        }
      }
      VMCNT0();
      __syncthreads();
      if (tid == 0) {
        unsigned old = __hip_atomic_fetch_add((unsigned*)(lws + OFF_CW), 1u, __ATOMIC_RELAXED, __HIP_MEMORY_SCOPE_AGENT);
        if (old == call * NGEMM - 1) {
          __builtin_amdgcn_fence(__ATOMIC_RELEASE, "");
          VMCNT0();
          #pragma unroll
          for (int e = 1; e < 8; ++e) st_sys(P.wsx[e] + OFF_WRDY + e * 128, call);
        }
      }
    }
    STAMP(2);

    const int xcd = bid & 7, slot = bid >> 3;
    const int wm = wave >> 1, wn = wave & 1;
    int tcount = 0;
    for (int u = slot; u < 64; u += NGEMM / 8, ++tcount) {
      const int c = xcd + 8 * (u >> 2), rb = u & 3;
      const int j = c % 7, f = j + 1;
      const int dj = dir_of_f(f);
      const int dq = (rb < 2) ? dj : 0;
      const int h = rb & 1;
      const int e0 = plane_elem(f, 0), e1 = plane_elem(f, 1), e2 = plane_elem(f, 2);
      if (wave == 0) {
        const char* word; unsigned target;
        int l = lane & 7;
        if (l < 3) { word = lws + OFF_WCNT + (c * 3 + l) * 128; target = call * 32u; }
        else if (l < 6) { int e = (l == 3) ? e0 : (l == 4) ? e1 : e2; word = lws + OFF_XCNT + (e * 8 + dq) * 128; target = call * 128u; }
        else if (l == 6) { word = lws + OFF_CX; target = call * NWG; }
        else { word = lws + OFF_CW; target = call * NGEMM; }
        (void)wave_wait_s<GSLEEP>(word, target, tmo, call);
        __builtin_amdgcn_fence(__ATOMIC_ACQUIRE, "agent");
        VMCNT0();
      }
      __syncthreads();
      STAMP(3 + tcount * 3);

      const char* a0 = lws + OFF_XC + par * 4 * MiB + (size_t)(((rank ^ dq) * 2 + h)) * 262144;
      const char* a1 = lws + OFF_XP + (size_t)(e0 * 8 + dq) * 524288 + h * 262144;
      const char* a2 = lws + OFF_XP + (size_t)(e1 * 8 + dq) * 524288 + h * 262144;
      const char* a3 = lws + OFF_XP + (size_t)(e2 * 8 + dq) * 524288 + h * 262144;
      const char* b0 = lws + OFF_WC + par * 16 * MiB + (size_t)c * 131072;
      const char* b1 = lws + OFF_WP + (size_t)(c * 3 + 0) * 131072;
      const char* b2 = lws + OFF_WP + (size_t)(c * 3 + 1) * 131072;
      const char* b3 = lws + OFF_WP + (size_t)(c * 3 + 2) * 131072;

      f4 acc[4][4];
      #pragma unroll
      for (int mi = 0; mi < 4; ++mi)
        #pragma unroll
        for (int ni = 0; ni < 4; ++ni) acc[mi][ni] = f4{0.f, 0.f, 0.f, 0.f};

      gemm_issue(smem, 0, 0, a0, a1, a2, a3, b0, b1, b2, b3, wave, lane);
      gemm_issue(smem, 1, 1, a0, a1, a2, a3, b0, b1, b2, b3, wave, lane);
      int buf = 0;
      for (int t = 0; t < 32; ++t) {
        if (t < 31) asm volatile("s_waitcnt vmcnt(6)" ::: "memory");
        else asm volatile("s_waitcnt vmcnt(0)" ::: "memory");
        asm volatile("s_waitcnt lgkmcnt(0)" ::: "memory");
        __builtin_amdgcn_s_barrier();
        int nbuf = buf + 2; if (nbuf >= 3) nbuf -= 3;
        if (t + 2 < 32) gemm_issue(smem, t + 2, nbuf, a0, a1, a2, a3, b0, b1, b2, b3, wave, lane);
        const char* sa = smem + buf * STAGE_B + wm * 4096 + lane * 16;
        const char* sb = smem + buf * STAGE_B + 32768 + wn * 4096 + lane * 16;
        #pragma unroll
        for (int kb = 0; kb < 2; ++kb) {
          h8 af[4], bf[4];
          #pragma unroll
          for (int i = 0; i < 4; ++i) af[i] = *(const h8*)(sa + kb * 16384 + i * 1024);
          #pragma unroll
          for (int i = 0; i < 4; ++i) bf[i] = *(const h8*)(sb + kb * 8192 + i * 1024);
          #pragma unroll
          for (int mi = 0; mi < 4; ++mi)
            #pragma unroll
            for (int ni = 0; ni < 4; ++ni)
              acc[mi][ni] = __builtin_amdgcn_mfma_f32_16x16x32_f16(bf[ni], af[mi], acc[mi][ni], 0, 0, 0);
        }
        buf = buf + 1; if (buf >= 3) buf = 0;
      }
      STAMP(4 + tcount * 3);
      {
        int dst = (rb < 2) ? (int)(OFF_SP + par * 32 * MiB) : (int)OFF_SO;
        dst += (c * 2 + h) * 131072 + (wm * 64 + (lane & 15)) * 512 + wn * 256 + (lane >> 4) * 16;
        #pragma unroll
        for (int mi = 0; mi < 4; ++mi)
          #pragma unroll
          for (int ni = 0; ni < 4; ++ni)
            __builtin_amdgcn_raw_buffer_store_b128(__builtin_bit_cast(u4, acc[mi][ni]), Lr, dst + mi * 8192 + ni * 64, 0, 17);
        VMCNT0();
        __syncthreads();
        if (tid == 0) {
          int fu = (j * 64 + (c / 7) * 2 + h) * 4;
          if (rb < 2) st_sys(peer_ws(P, dj) + OFF_FLC + fu, call);
          else st_sys(lws + OFF_FLO + fu, call);
        }
      }
      STAMP(5 + tcount * 3);
    }
  } else if (bid < NGEMM + 28) {
    const int pw = bid - NGEMM;
    const int e = (pw % 7) + 1;
    char* const rws = peer_ws(P, e);
    const int j0 = plane_elem(e, 0) - 1, j1 = plane_elem(e, 1) - 1, j2 = plane_elem(e, 2) - 1;
    const int nW = 54 + (j0 < 2) + (j1 < 2);
    const int jc = f_of_dir(e) - 1;
    const int ntile = ((NCT - jc + 6) / 7) * 2;
    const int TX = 512, TW = TX + nW * 32, TT = TW + ntile * 32;
    const unsigned qbase = (call - 1) * (unsigned)(TT + PWAVES);
    const unsigned qoff = OFF_Q + e * 128;
    const unsigned lo16 = lane * 16;
    if (wave < PW_PER_WG) {
    unsigned tv = 0;
    DRAW();
    bool ok = wave_wait(lws + OFF_XRDY + e * 128, call, tmo, call);
    STAMP(2);
    asm volatile("s_waitcnt vmcnt(0)" : "+v"(tv) :: "memory");
    int t = (int)(__builtin_amdgcn_readfirstlane(tv) - qbase);
    u4 ra0, ra1, rb0, rb1;

#define CDESC(t, SO, LO, CN) do { \
      if ((t) < TX) { \
        int b_ = (t) >> 7; \
        int dq_ = (b_ == 0) ? 0 : dir_of_f(plane_elem(e, b_ - 1)); \
        SO = (unsigned)(OFF_XC + par * 4 * MiB) + (rank ^ dq_) * 524288 + ((t) & 127) * 4096; \
        LO = (unsigned)OFF_XP + (e * 8 + dq_) * 524288 + ((t) & 127) * 4096; \
        CN = OFF_XCNT + (e * 8 + dq_) * 128; \
      } else { \
        if (!wseen) { wseen = true; STAMP(3); ok = ok && wave_wait_nl(lws + OFF_WRDY + e * 128, call, tmo, call); } \
        int idx_ = ((t) - TX) >> 5, g_ = idx_ / 3, pos_ = idx_ - g_ * 3; \
        int jj_ = (pos_ == 0) ? j0 : (pos_ == 1) ? j1 : j2; \
        int c_ = g_ * 7 + jj_; \
        int i_ = (plane_elem(jj_ + 1, 0) == e) ? 0 : (plane_elem(jj_ + 1, 1) == e) ? 1 : 2; \
        SO = (unsigned)(OFF_WC + par * 16 * MiB) + c_ * 131072 + ((t) & 31) * 4096; \
        LO = (unsigned)OFF_WP + (c_ * 3 + i_) * 131072 + ((t) & 31) * 4096; \
        CN = OFF_WCNT + (c_ * 3 + i_) * 128; \
      } } while (0)
    if (ok && t < TW) {
      bool wseen = false;
      unsigned pending = 0;
      unsigned soff, loff, cnt, nsoff = 0, nloff = 0, ncnt = 0;
      CDESC(t, soff, loff, cnt);
      GLOAD2(ra0, ra1, rws, soff + lo16, 0);
      DRAW();
      asm volatile("s_waitcnt vmcnt(0)" ::: "memory");
      for (;;) {
        GLOAD2(rb0, rb1, rws, soff + lo16, 2048);
        asm volatile("s_waitcnt vmcnt(4)" : "+v"(tv) :: "memory");
        GSTORE2(ra0, ra1, lws, loff + lo16, 0);
        t = (int)(__builtin_amdgcn_readfirstlane(tv) - qbase);
        const bool more = t < TW;
        if (more) {
          CDESC(t, nsoff, nloff, ncnt);
          GLOAD2(ra0, ra1, rws, nsoff + lo16, 0);
          DRAW();
          asm volatile("s_waitcnt vmcnt(5)" ::: "memory");
        } else {
          asm volatile("s_waitcnt vmcnt(2)" ::: "memory");
        }
        if (pending) SIGNAL(pending);
        pending = cnt;
        GSTORE2(rb0, rb1, lws, loff + lo16, 2048);
        if (!more || !ok) break;
        soff = nsoff; loff = nloff; cnt = ncnt;
      }
      asm volatile("s_waitcnt vmcnt(0)" : "+v"(tv) :: "memory");
      if (ok) SIGNAL(pending);
      else {
        while (t < TW) { t = (int)(__builtin_amdgcn_readfirstlane(tv) - qbase); if (t < TW) { DRAW(); asm volatile("s_waitcnt vmcnt(0)" : "+v"(tv) :: "memory"); } }
      }
    } else if (!ok) {
      while (t < TW) { DRAW(); asm volatile("s_waitcnt vmcnt(0)" : "+v"(tv) :: "memory"); t = (int)(__builtin_amdgcn_readfirstlane(tv) - qbase); }
    }
    STAMP(4);
    {
      int ready = 0;
      const unsigned olane = ((lane >> 5) * NCOL + (lane & 31) * 4) * 4;
      u4 la0, la1, lb0, lb1;
      unsigned toff = 0, ooff = 0, ntoff = 0, nooff = 0;
      const char* const rsp = rws + OFF_SP + par * 32 * MiB;
      const char* const lso = lws + OFF_SO;
      char* const outp = (char*)P.out;
#define RDESC(t, TO, OO) do { \
        const int u_ = ((t) - TW) >> 5, wv_ = ((t) - TW) & 31; \
        const int c_ = jc + 7 * (u_ >> 1), h_ = u_ & 1; \
        TO = (c_ * 2 + h_) * 131072 + wv_ * 4096; \
        OO = ((h_ * 256 + wv_ * 8) * NCOL + c_ * BN) * 4; \
      } while (0)
#define POLLC(t) do { \
      const int u_ = ((t) - TW) >> 5; \
      unsigned long long t0_ = __builtin_amdgcn_s_memrealtime(); \
      for (;;) { \
        int uu_ = u_ + lane; if (uu_ > 63) uu_ = 63; \
        unsigned a_ = ld_sys(lws + OFF_FLC + (jc * 64 + uu_) * 4); \
        unsigned b_ = ld_sys(lws + OFF_FLO + (jc * 64 + uu_) * 4); \
        bool good_ = ((int)(a_ - call) >= 0) && ((int)(b_ - call) >= 0) && (u_ + lane < ntile); \
        unsigned long long m_ = __ballot(good_); \
        int n_ = (~m_ == 0ull) ? 64 : __builtin_ctzll(~m_); \
        if (n_ > 0) { ready = u_ + n_; break; } \
        if (ld_agent(tmo) == call || __builtin_amdgcn_s_memrealtime() - t0_ > DEADLINE_TICKS) { \
          __hip_atomic_store((unsigned*)tmo, call, __ATOMIC_RELAXED, __HIP_MEMORY_SCOPE_AGENT); \
          ok = false; break; \
        } \
        __builtin_amdgcn_s_sleep(2); \
      } \
      asm volatile("s_waitcnt vmcnt(0)" ::: "memory"); } while (0)
#define RFIN(R0, R1, L0, L1, IMM) do { \
        f4 r0_ = __builtin_bit_cast(f4, R0) + __builtin_bit_cast(f4, L0); \
        f4 r1_ = __builtin_bit_cast(f4, R1) + __builtin_bit_cast(f4, L1); \
        _Pragma("unroll") for (int k_ = 0; k_ < 4; ++k_) { r0_[k_] = fmaxf(r0_[k_], 0.f); r1_[k_] = fmaxf(r1_[k_], 0.f); } \
        asm volatile("global_store_dwordx4 %0, %1, %4\n\tglobal_store_dwordx4 %3, %2, %4\n\ts_nop 1" \
                     :: "v"(ooff + olane + (IMM) * 4 * NCOL * 4), "v"(r0_), "v"(r1_), "v"(ooff + olane + ((IMM) * 4 + 2) * NCOL * 4), "s"(outp) : "memory"); \
      } while (0)
      while (ok && t < TT) {
        if (((t - TW) >> 5) >= ready) { POLLC(t); if (!ok) break; }
        RDESC(t, toff, ooff);
        GLOAD2(ra0, ra1, rsp, toff + lo16, 0);
        GLOAD2(la0, la1, lso, toff + lo16, 0);
        DRAW();
        asm volatile("s_waitcnt vmcnt(0)" ::: "memory");
        for (;;) {
          GLOAD2(rb0, rb1, rsp, toff + lo16, 2048);
          GLOAD2(lb0, lb1, lso, toff + lo16, 2048);
          asm volatile("s_waitcnt vmcnt(6)" : "+v"(ra0), "+v"(ra1), "+v"(la0), "+v"(la1), "+v"(tv) :: "memory");
          RFIN(ra0, ra1, la0, la1, 0);
          t = (int)(__builtin_amdgcn_readfirstlane(tv) - qbase);
          const bool more = (t < TT) && (((t - TW) >> 5) < ready);
          if (more) {
            RDESC(t, ntoff, nooff);
            GLOAD2(ra0, ra1, rsp, ntoff + lo16, 0);
            GLOAD2(la0, la1, lso, ntoff + lo16, 0);
            DRAW();
            asm volatile("s_waitcnt vmcnt(7)" ::: "memory");
          } else {
            asm volatile("s_waitcnt vmcnt(2)" ::: "memory");
          }
          asm volatile("" : "+v"(rb0), "+v"(rb1), "+v"(lb0), "+v"(lb1) :: "memory");
          RFIN(rb0, rb1, lb0, lb1, 1);
          if (!more) break;
          toff = ntoff; ooff = nooff;
        }
      }
      while (t < TT) {
        RDESC(t, toff, ooff);
        float nv = __builtin_nanf("");
        f4 q = f4{nv, nv, nv, nv};
        #pragma unroll
        for (int k = 0; k < 4; ++k) *(f4*)(outp + ooff + olane + k * 2 * NCOL * 4) = q;
        DRAW();
        asm volatile("s_waitcnt vmcnt(0)" : "+v"(tv) :: "memory");
        t = (int)(__builtin_amdgcn_readfirstlane(tv) - qbase);
      }
    }
    }
    STAMP(5);
  }
}

}

extern "C" void kernel_launch(int rank, int nranks, ncclComm_t comm,
                              hipStream_t stream,
                              void* const* d_in, const int* in_sizes, int n_in,
                              void* d_out, int out_size,
                              const int* in_dtypes, int out_dtype,
                              void* const* d_out_all,
                              void* const* d_ws_all, size_t ws_bytes) {
  (void)comm; (void)in_sizes; (void)n_in; (void)out_size; (void)in_dtypes; (void)out_dtype; (void)d_out_all;
  if (nranks != NRANK || ws_bytes < WS_NEED) {
    fprintf(stderr, "unsupported configuration: nranks %d ws %zu\n", nranks, ws_bytes);
    return;
  }
  HIP_CHECK(hipFuncSetAttribute((const void*)fused_gemm_rs, hipFuncAttributeMaxDynamicSharedMemorySize, LDS_BYTES));
  Params P;
  P.x = (const float*)d_in[0];
  P.w = (const float*)d_in[1];
  P.out = (float*)d_out;
  for (int e = 0; e < NRANK; ++e) P.wsx[e] = (char*)d_ws_all[rank ^ e];
  P.rank = rank;
  fused_gemm_rs<<<NWG, NTHR, LDS_BYTES, stream>>>(P);
  HIP_CHECK(hipGetLastError());
}
```

```cpp
#include <hip/hip_runtime.h>
#include <rccl/rccl.h>
#include <cstdio>
#include <cstdint>

#define HIP_CHECK(expr) do { hipError_t _e = (expr); if (_e != hipSuccess) { \
    fprintf(stderr, "HIP error %s at %s:%d\n", hipGetErrorString(_e), __FILE__, __LINE__); \
    return; } } while (0)

namespace {

typedef _Float16 h8 __attribute__((ext_vector_type(8)));
typedef float f4 __attribute__((ext_vector_type(4)));
typedef unsigned u4 __attribute__((ext_vector_type(4)));

constexpr int NRANK = 8;
constexpr int NCOL = 16384;
constexpr int KS = 512;
constexpr int BN = 128;
constexpr int NCT = NCOL / BN;
constexpr int NWG = 256;
constexpr int NGEMM = 224;
constexpr int NTHR = 512;
#define PW_PER_WG 8
constexpr int PWAVES = 4 * PW_PER_WG;

constexpr size_t MiB = 1024 * 1024;
constexpr unsigned OFF_TICKET = 0;
constexpr unsigned OFF_TMO = 128;
constexpr unsigned OFF_CX = 256;
constexpr unsigned OFF_CW = 384;
constexpr unsigned OFF_XRDY = 1024;
constexpr unsigned OFF_WRDY = 2048;
constexpr unsigned OFF_Q = 3072;
constexpr unsigned OFF_XCNT = 8192;
constexpr unsigned OFF_WCNT = 32768;
constexpr unsigned OFF_FLC = 65536;
constexpr unsigned OFF_FLO = 69632;
constexpr unsigned OFF_DBG = 131072;
constexpr size_t OFF_WC = 1 * MiB;
constexpr size_t OFF_XC = OFF_WC + 32 * MiB;
constexpr size_t OFF_WP = OFF_XC + 8 * MiB;
constexpr size_t OFF_XP = OFF_WP + 16 * MiB;
constexpr size_t OFF_SO = OFF_XP + 32 * MiB;
constexpr size_t OFF_SP = OFF_SO + 32 * MiB;
constexpr size_t WS_NEED = OFF_SP + 64 * MiB;

constexpr unsigned STAGE_B = 49152;
constexpr unsigned LDS_BC = 3 * STAGE_B;
constexpr unsigned LDS_BYTES = LDS_BC + 64;

constexpr unsigned long long DEADLINE_TICKS = 50000000ull;

constexpr int U_X1 = 0;
constexpr int U_W = 192;
constexpr int U_X2 = 704;
constexpr int U_XO = 896;
constexpr int U_X3 = 1024;
constexpr int U_C = 1408;
constexpr int U_END = 2432;

struct Params {
  const float* x;
  const float* w;
  float* out;
  char* wsx[NRANK];
  int rank;
};
__device__ __forceinline__ char* peer_ws(const Params& P, int e) {
  char* p = P.wsx[1];
  p = (e == 2) ? P.wsx[2] : p; p = (e == 3) ? P.wsx[3] : p; p = (e == 4) ? P.wsx[4] : p;
  p = (e == 5) ? P.wsx[5] : p; p = (e == 6) ? P.wsx[6] : p; p = (e == 7) ? P.wsx[7] : p;
  return p;
}

__device__ __forceinline__ unsigned ld_agent(const char* p) {
  return __hip_atomic_load((const unsigned*)p, __ATOMIC_RELAXED, __HIP_MEMORY_SCOPE_AGENT);
}
__device__ __forceinline__ unsigned ld_sys(const char* p) {
  return __hip_atomic_load((const unsigned*)p, __ATOMIC_RELAXED, __HIP_MEMORY_SCOPE_SYSTEM);
}
__device__ __forceinline__ void st_sys(char* p, unsigned v) {
  __hip_atomic_store((unsigned*)p, v, __ATOMIC_RELAXED, __HIP_MEMORY_SCOPE_SYSTEM);
}
#define VMCNT0() asm volatile("s_waitcnt vmcnt(0)" ::: "memory")

template <int SLEEP>
__device__ __forceinline__ bool wave_wait(const char* word, unsigned target, char* tmo, unsigned call) {
  unsigned v = ld_sys(word);
  if (__all((int)(v - target) >= 0)) return true;
  unsigned long long t0 = __builtin_amdgcn_s_memrealtime();
  for (;;) {
    __builtin_amdgcn_s_sleep(SLEEP);
    v = ld_sys(word);
    if (__all((int)(v - target) >= 0)) return true;
    if (ld_agent(tmo) == call || __builtin_amdgcn_s_memrealtime() - t0 > DEADLINE_TICKS) {
      __hip_atomic_store((unsigned*)tmo, call, __ATOMIC_RELAXED, __HIP_MEMORY_SCOPE_AGENT);
      return false;
    }
  }
}
#define WCAUX 17
#define GSLEEP 32

__device__ __forceinline__ unsigned pack2(float a, float b) {
  typedef _Float16 h2 __attribute__((ext_vector_type(2)));
  h2 v = {(_Float16)a, (_Float16)b};
  return __builtin_bit_cast(unsigned, v);
}

#define GLOAD2(D0, D1, BASE, VOFF, IMM) \
  asm volatile("global_load_dwordx4 %0, %2, %3 offset:%4 sc0 sc1\n\tglobal_load_dwordx4 %1, %2, %3 offset:%5 sc0 sc1" \
               : "=&v"(D0), "=&v"(D1) : "v"(VOFF), "s"(BASE), "i"(IMM), "i"(IMM + 1024) : "memory")
#define GSTORE2(D0, D1, BASE, VOFF, IMM) \
  asm volatile("global_store_dwordx4 %2, %0, %3 offset:%4 sc0 sc1\n\tglobal_store_dwordx4 %2, %1, %3 offset:%5 sc0 sc1\n\ts_nop 1" \
               :: "v"(D0), "v"(D1), "v"(VOFF), "s"(BASE), "i"(IMM), "i"(IMM + 1024) : "memory")
#define DRAW() do { if (lane == 0) asm volatile("global_atomic_add %0, %1, %2, %3 sc0" : "+v"(tv) : "v"(qoff), "v"(1u), "s"(lws) : "memory"); } while (0)
#define SIGNAL(OFFS) do { if (lane == 0) asm volatile("global_atomic_add %0, %1, %2" :: "v"(OFFS), "v"(1u), "s"(lws) : "memory"); } while (0)

#define STAMP(slot) do {} while (0)

__device__ __forceinline__ void gemm_tile(f4 (&acc)[4][4], char* smem, char* lws, char* tmo, unsigned call, int nsh,
                                          const char* a_loc, const char* b_loc, int e_first, int xsel, int ct,
                                          int wave, int lane) {
  const int wm = wave >> 1, wn = wave & 1;
  #pragma unroll
  for (int mi = 0; mi < 4; ++mi)
    #pragma unroll
    for (int ni = 0; ni < 4; ++ni) acc[mi][ni] = f4{0.f, 0.f, 0.f, 0.f};
  const int nt = nsh * 8;
  const char* ap = a_loc + wave * 4096 + lane * 16;
  const char* bp = b_loc + wave * 2048 + lane * 16;
#define ISSUE_STAGE(BUF) do { \
      char* dst_ = smem + (BUF) * STAGE_B; \
      _Pragma("unroll") for (int i = 0; i < 4; ++i) \
        __builtin_amdgcn_global_load_lds((const unsigned*)(ap + i * 1024), (unsigned*)(dst_ + wave * 4096 + i * 1024), 16, 0, 16); \
      _Pragma("unroll") for (int i = 0; i < 2; ++i) \
        __builtin_amdgcn_global_load_lds((const unsigned*)(bp + i * 1024), (unsigned*)(dst_ + 32768 + wave * 2048 + i * 1024), 16, 0, 16); \
      ap += 32768; bp += 16384; } while (0)
#define READ_FRAGS(AF, BF, BUF, KB) do { \
      const char* sa_ = smem + (BUF) * STAGE_B + (KB) * 16384 + wm * 4096 + lane * 16; \
      const char* sb_ = smem + (BUF) * STAGE_B + 32768 + (KB) * 8192 + wn * 4096 + lane * 16; \
      _Pragma("unroll") for (int i = 0; i < 4; ++i) { BF[i] = *(const h8*)(sb_ + i * 1024); AF[i] = *(const h8*)(sa_ + i * 1024); } } while (0)
#define MMA(AF, BF) do { \
      __builtin_amdgcn_s_setprio(1); \
      _Pragma("unroll") for (int mi = 0; mi < 4; ++mi) \
        _Pragma("unroll") for (int ni = 0; ni < 4; ++ni) \
          acc[mi][ni] = __builtin_amdgcn_mfma_f32_16x16x32_f16(BF[ni], AF[mi], acc[mi][ni], 0, 0, 0); \
      __builtin_amdgcn_s_setprio(0); } while (0)
  h8 a0[4], b0[4], a1[4], b1[4];
  ISSUE_STAGE(0); ISSUE_STAGE(1); ISSUE_STAGE(2);
  asm volatile("s_waitcnt vmcnt(12)" ::: "memory");
  __builtin_amdgcn_s_barrier();
  READ_FRAGS(a0, b0, 0, 0);
  int buf = 0;
  for (int t = 0; t < nt; ++t) {
    READ_FRAGS(a1, b1, buf, 1);
    __builtin_amdgcn_sched_barrier(0);
    MMA(a0, b0);
    __builtin_amdgcn_sched_barrier(0);
    const int nbuf = (buf == 2) ? 0 : buf + 1;
    if (t + 1 < nt) {
      const int ti = t + 3;
      const bool next_shard = ti < nt && (ti & 7) == 0;
      int e = e_first - (ti >> 3); e += (e < 0) ? 7 : 0; e += 1;
      if (next_shard && wave == 0) {
        const char* word = (lane & 1) ? (lws + OFF_WCNT + (e * 16 + ct) * 128) : (lws + OFF_XCNT + (e * 16 + xsel) * 128);
        unsigned target = (lane & 1) ? call * 32u : call * 64u;
        (void)wave_wait<GSLEEP>(word, target, tmo, call);
      }
      if (t + 2 < nt) asm volatile("s_waitcnt vmcnt(6) lgkmcnt(0)" ::: "memory");
      else asm volatile("s_waitcnt vmcnt(0) lgkmcnt(0)" ::: "memory");
      __builtin_amdgcn_s_barrier();
      if (ti < nt) {
        if (next_shard) {
          ap = lws + OFF_XP + (size_t)(e * 16 + xsel) * 262144 + wave * 4096 + lane * 16;
          bp = lws + OFF_WP + (size_t)(e * 16 + ct) * 131072 + wave * 2048 + lane * 16;
        }
        ISSUE_STAGE(buf);
      }
      READ_FRAGS(a0, b0, nbuf, 0);
    }
    __builtin_amdgcn_sched_barrier(0);
    MMA(a1, b1);
    __builtin_amdgcn_sched_barrier(0);
    buf = nbuf;
  }
}

__global__ void __launch_bounds__(NTHR, 2) fused_gemm_rs(Params P) {
  extern __shared__ __attribute__((aligned(16))) char smem[];
  const int tid = threadIdx.x, lane = tid & 63;
  const int wave = __builtin_amdgcn_readfirstlane(tid >> 6);
  const int bid = blockIdx.x;
  const int rank = P.rank;
  char* const lws = P.wsx[0];
  char* const tmo = lws + OFF_TMO;

  STAMP(0);
  if (tid == 0) {
    unsigned t = __hip_atomic_fetch_add((unsigned*)(lws + OFF_TICKET), 1u, __ATOMIC_RELAXED, __HIP_MEMORY_SCOPE_AGENT);
    *(volatile unsigned*)(smem + LDS_BC) = t / NWG + 1;
  }
  f4 xv[2][2];
  {
    #pragma unroll
    for (int i = 0; i < 2; ++i) {
      int F = (bid * 8 + wave) * 2 + i;
      int rh = F >> 8, kb = (F >> 4) & 15, rb = F & 15;
      const float* src = P.x + (size_t)(rh * 256 + rb * 16 + (lane & 15)) * KS + kb * 32 + (lane >> 4) * 8;
      xv[i][0] = *(const f4*)src;
      xv[i][1] = *(const f4*)(src + 4);
    }
  }
  __syncthreads();
  const unsigned call = __builtin_amdgcn_readfirstlane(*(volatile unsigned*)(smem + LDS_BC));
  const unsigned par = call & 1;

  auto Lr = __builtin_amdgcn_make_buffer_rsrc((void*)lws, 0, (int)WS_NEED, 0x00020000);

  {
    #pragma unroll
    for (int i = 0; i < 2; ++i) {
      int F = (bid * 8 + wave) * 2 + i;
      u4 o;
      o[0] = pack2(xv[i][0][0], xv[i][0][1]);
      o[1] = pack2(xv[i][0][2], xv[i][0][3]);
      o[2] = pack2(xv[i][1][0], xv[i][1][1]);
      o[3] = pack2(xv[i][1][2], xv[i][1][3]);
      __builtin_amdgcn_raw_buffer_store_b128(o, Lr, (int)(OFF_XC + par * 4 * MiB) + F * 1024 + lane * 16, 0, 17);
    }
    VMCNT0();
    __syncthreads();
    if (tid == 0) {
      unsigned old = __hip_atomic_fetch_add((unsigned*)(lws + OFF_CX), 1u, __ATOMIC_RELAXED, __HIP_MEMORY_SCOPE_AGENT);
      if (old == call * NWG - 1) {
        __builtin_amdgcn_fence(__ATOMIC_RELEASE, "");
        VMCNT0();
        #pragma unroll
        for (int e = 1; e < 8; ++e) st_sys(P.wsx[e] + OFF_XRDY + e * 128, call);
      }
    }
  }
  STAMP(1);

  if (bid < NGEMM) {
    {
      const int T0 = bid * 8 + wave;
      const bool third = T0 + 2 * NGEMM * 8 < 4096;
      float v[3][4][8];
      #pragma unroll
      for (int r = 0; r < 3; ++r) {
        if (r == 2 && !third) break;
        const int T = T0 + r * NGEMM * 8;
        const float* src = P.w + (size_t)((T >> 6) * 8) * NCOL + (T & 63) * 256 + lane;
        #pragma unroll
        for (int j = 0; j < 8; ++j)
          #pragma unroll
          for (int i = 0; i < 4; ++i) v[r][i][j] = src[(size_t)j * NCOL + i * 64];
      }
      #pragma unroll
      for (int r = 0; r < 3; ++r) {
        if (r == 2 && !third) break;
        const int T = T0 + r * NGEMM * 8;
        const int cg = T & 63, ko = T >> 6;
        int dst = (int)(OFF_WC + par * 16 * MiB) + cg * 2 * 131072 + (ko >> 2) * 8192 + (lane >> 4) * 1024 + ((ko & 3) * 16 + (lane & 15)) * 16;
        #pragma unroll
        for (int i = 0; i < 4; ++i) {
          u4 o;
          o[0] = pack2(v[r][i][0], v[r][i][1]);
          o[1] = pack2(v[r][i][2], v[r][i][3]);
          o[2] = pack2(v[r][i][4], v[r][i][5]);
          o[3] = pack2(v[r][i][6], v[r][i][7]);
          __builtin_amdgcn_raw_buffer_store_b128(o, Lr, dst + (i >> 1) * 131072 + (i & 1) * 4096, 0, WCAUX);
        }
      }
      VMCNT0();
      __syncthreads();
      if (tid == 0) {
        unsigned old = __hip_atomic_fetch_add((unsigned*)(lws + OFF_CW), 1u, __ATOMIC_RELAXED, __HIP_MEMORY_SCOPE_AGENT);
        if (old == call * NGEMM - 1) {
          __builtin_amdgcn_fence(__ATOMIC_RELEASE, "");
          VMCNT0();
          #pragma unroll
          for (int e = 1; e < 8; ++e) st_sys(P.wsx[e] + OFF_WRDY + e * 128, call);
        }
      }
    }
    STAMP(2);
    {
      const char* word = (lane & 1) ? (lws + OFF_CW) : (lws + OFF_CX);
      unsigned target = (lane & 1) ? call * NGEMM : call * NWG;
      if (wave == 0) (void)wave_wait<GSLEEP>(word, target, tmo, call);
      __syncthreads();
    }
    STAMP(3);

    const int xcd = bid & 7, slot = bid >> 3;
    const int ct = 4 * (xcd >> 1) + (slot & 3);
    const int h = xcd & 1, dq = (slot >> 2) + 1;
    const int wm = wave >> 1, wn = wave & 1;
    const char* const xc = lws + OFF_XC + par * 4 * MiB;
    const char* const wc = lws + OFF_WC + par * 16 * MiB;
    const int slane = (wm * 64 + (lane & 15)) * 512 + wn * 256 + (lane >> 4) * 16;
    f4 acc[4][4];

    for (int jb = 0; jb < 3; ++jb) {
      const int job = (jb == 0) ? 0 : 3 - jb;
      if (job == 1 && !(h == 0 && dq <= 2)) continue;
      int nsh, e_first, xsel;
      const char *a_loc, *b_loc;
      if (job == 0) {
        nsh = 1; e_first = 0; xsel = 0;
        a_loc = xc + (size_t)(rank * 2 + h) * 262144;
        b_loc = wc + (size_t)((rank ^ dq) * 16 + ct) * 131072;
      } else if (job == 1) {
        nsh = 8; e_first = 7; xsel = dq - 1;
        a_loc = xc + (size_t)(rank * 2 + dq - 1) * 262144;
        b_loc = wc + (size_t)(rank * 16 + ct) * 131072;
      } else {
        nsh = 7; e_first = dq - 1; xsel = dq * 2 + h;
        a_loc = xc + (size_t)((rank ^ dq) * 2 + h) * 262144;
        b_loc = wc + (size_t)(rank * 16 + ct) * 131072;
      }
      gemm_tile(acc, smem, lws, tmo, call, nsh, a_loc, b_loc, e_first, xsel, ct, wave, lane);
      STAMP(4 + job * 2);
      if (job == 1) {
        const bool bad = ld_agent(tmo) == call;
        float* o = P.out + (size_t)((dq - 1) * 256 + wm * 64 + (lane & 15)) * NCOL + (rank * 16 + ct) * BN + wn * 64 + (lane >> 4) * 4;
        #pragma unroll
        for (int mi = 0; mi < 4; ++mi)
          #pragma unroll
          for (int ni = 0; ni < 4; ++ni) {
            f4 v = acc[mi][ni];
            #pragma unroll
            for (int k = 0; k < 4; ++k) v[k] = bad ? __builtin_nanf("") : fmaxf(v[k], 0.f);
            *(f4*)(o + (size_t)mi * 16 * NCOL + ni * 16) = v;
          }
        __syncthreads();
      } else {
        int dst = (job == 0) ? (int)OFF_SO : (int)(OFF_SP + par * 32 * MiB);
        dst += ((dq * 2 + h) * 16 + ct) * 131072 + slane;
        #pragma unroll
        for (int mi = 0; mi < 4; ++mi)
          #pragma unroll
          for (int ni = 0; ni < 4; ++ni)
            __builtin_amdgcn_raw_buffer_store_b128(__builtin_bit_cast(u4, acc[mi][ni]), Lr, dst + mi * 8192 + ni * 64, 0, 17);
        VMCNT0();
        __syncthreads();
        if (tid == 0) {
          int fu = (dq * 32 + h * 16 + ct) * 4;
          if (job == 0) st_sys(lws + OFF_FLO + fu, call);
          else st_sys(peer_ws(P, dq) + OFF_FLC + fu, call);
        }
      }
      STAMP(5 + job * 2);
    }
  } else if (bid < NGEMM + 28) {
    const int pw = bid - NGEMM;
    const int e = (pw % 7) + 1;
    char* const rws = peer_ws(P, e);
    const unsigned qbase = (call - 1) * (unsigned)(U_END + PWAVES);
    const unsigned qoff = OFF_Q + e * 128;
    const unsigned lo16 = lane * 16;
    if (wave < PW_PER_WG) {
    unsigned tv = 0;
    DRAW();
    bool ok = wave_wait<2>(lws + OFF_XRDY + e * 128, call, tmo, call);
    STAMP(2);
    asm volatile("s_waitcnt vmcnt(0)" : "+v"(tv) :: "memory");
    int t = (int)(__builtin_amdgcn_readfirstlane(tv) - qbase);
    u4 ra0, ra1, rb0, rb1;

#define CDESC(t, SO, LO, CN) do { \
      if ((t) >= U_W && (t) < U_X2) { \
        if (!wseen) { wseen = true; STAMP(3); ok = ok && wave_wait<2>(lws + OFF_WRDY + e * 128, call, tmo, call); } \
        int ct_ = ((t) - U_W) >> 5; \
        SO = (unsigned)(OFF_WC + par * 16 * MiB) + (rank * 16 + ct_) * 131072 + ((t) & 31) * 4096; \
        LO = (unsigned)OFF_WP + (e * 16 + ct_) * 131072 + ((t) & 31) * 4096; \
        CN = OFF_WCNT + (e * 16 + ct_) * 128; \
      } else { \
        int dq_, h_; \
        if ((t) >= U_XO && (t) < U_X3) { dq_ = 0; h_ = ((t) - U_XO) >> 6; } \
        else { \
          int k_ = ((t) < U_W) ? 1 + ((t) >> 6) : ((t) < U_XO) ? 4 + (((t) - U_X2) >> 6) : 1 + (((t) - U_X3) >> 6); \
          h_ = ((t) >= U_X3) ? 1 : 0; \
          dq_ = e - 1 + k_; dq_ -= (dq_ >= 7) ? 7 : 0; dq_ += 1; \
        } \
        SO = (unsigned)(OFF_XC + par * 4 * MiB) + ((rank ^ dq_) * 2 + h_) * 262144 + ((t) & 63) * 4096; \
        LO = (unsigned)OFF_XP + ((e * 8 + dq_) * 2 + h_) * 262144 + ((t) & 63) * 4096; \
        CN = OFF_XCNT + ((e * 8 + dq_) * 2 + h_) * 128; \
      } } while (0)
    if (ok && t < U_C) {
      bool wseen = false;
      unsigned pending = 0;
      unsigned soff, loff, cnt, nsoff = 0, nloff = 0, ncnt = 0;
      CDESC(t, soff, loff, cnt);
      GLOAD2(ra0, ra1, rws, soff + lo16, 0);
      DRAW();
      asm volatile("s_waitcnt vmcnt(0)" ::: "memory");
      for (;;) {
        GLOAD2(rb0, rb1, rws, soff + lo16, 2048);
        asm volatile("s_waitcnt vmcnt(4)" : "+v"(tv) :: "memory");
        GSTORE2(ra0, ra1, lws, loff + lo16, 0);
        t = (int)(__builtin_amdgcn_readfirstlane(tv) - qbase);
        const bool more = t < U_C;
        if (more) {
          CDESC(t, nsoff, nloff, ncnt);
          GLOAD2(ra0, ra1, rws, nsoff + lo16, 0);
          DRAW();
          asm volatile("s_waitcnt vmcnt(5)" ::: "memory");
        } else {
          asm volatile("s_waitcnt vmcnt(2)" ::: "memory");
        }
        if (pending) SIGNAL(pending);
        pending = cnt;
        GSTORE2(rb0, rb1, lws, loff + lo16, 2048);
        if (!more || !ok) break;
        soff = nsoff; loff = nloff; cnt = ncnt;
      }
      asm volatile("s_waitcnt vmcnt(0)" : "+v"(tv) :: "memory");
      if (ok) SIGNAL(pending);
      else {
        while (t < U_C) { t = (int)(__builtin_amdgcn_readfirstlane(tv) - qbase); if (t < U_C) { DRAW(); asm volatile("s_waitcnt vmcnt(0)" : "+v"(tv) :: "memory"); } }
      }
    } else if (!ok) {
      while (t < U_C) { DRAW(); asm volatile("s_waitcnt vmcnt(0)" : "+v"(tv) :: "memory"); t = (int)(__builtin_amdgcn_readfirstlane(tv) - qbase); }
    }
    STAMP(4);
    {
      int ready = 0;
      const unsigned olane = ((lane >> 5) * NCOL + (lane & 31) * 4) * 4;
      u4 la0, la1, lb0, lb1;
      unsigned toff = 0, ooff = 0, ntoff = 0, nooff = 0;
      const char* const rsp = rws + OFF_SP + par * 32 * MiB + (size_t)e * 32 * 131072;
      const char* const lso = lws + OFF_SO + (size_t)e * 32 * 131072;
      char* const outp = (char*)(P.out + (rank ^ e) * 2048);
#define RDESC(t, TO, OO) do { \
        const int u_ = ((t) - U_C) >> 5, wv_ = (t) & 31; \
        TO = ((t) - U_C) * 4096; \
        OO = (((u_ >> 4) * 256 + wv_ * 8) * NCOL + (u_ & 15) * BN) * 4; \
      } while (0)
#define POLLC(t) do { \
      const int u_ = ((t) - U_C) >> 5; \
      unsigned long long t0_ = __builtin_amdgcn_s_memrealtime(); \
      for (;;) { \
        int uu_ = u_ + lane; if (uu_ > 31) uu_ = 31; \
        unsigned a_ = ld_sys(lws + OFF_FLC + (e * 32 + uu_) * 4); \
        unsigned b_ = ld_sys(lws + OFF_FLO + (e * 32 + uu_) * 4); \
        bool good_ = ((int)(a_ - call) >= 0) && ((int)(b_ - call) >= 0) && (u_ + lane < 32); \
        unsigned long long m_ = __ballot(good_); \
        int n_ = __builtin_ctzll(~m_); \
        if (n_ > 0) { ready = u_ + n_; break; } \
        if (ld_agent(tmo) == call || __builtin_amdgcn_s_memrealtime() - t0_ > DEADLINE_TICKS) { \
          __hip_atomic_store((unsigned*)tmo, call, __ATOMIC_RELAXED, __HIP_MEMORY_SCOPE_AGENT); \
          ok = false; break; \
        } \
        __builtin_amdgcn_s_sleep(2); \
      } \
      asm volatile("s_waitcnt vmcnt(0)" ::: "memory"); } while (0)
#define RFIN(R0, R1, L0, L1, IMM) do { \
        f4 r0_ = __builtin_bit_cast(f4, R0) + __builtin_bit_cast(f4, L0); \
        f4 r1_ = __builtin_bit_cast(f4, R1) + __builtin_bit_cast(f4, L1); \
        _Pragma("unroll") for (int k_ = 0; k_ < 4; ++k_) { r0_[k_] = fmaxf(r0_[k_], 0.f); r1_[k_] = fmaxf(r1_[k_], 0.f); } \
        asm volatile("global_store_dwordx4 %0, %1, %4\n\tglobal_store_dwordx4 %3, %2, %4\n\ts_nop 1" \
                     :: "v"(ooff + olane + (IMM) * 4 * NCOL * 4), "v"(r0_), "v"(r1_), "v"(ooff + olane + ((IMM) * 4 + 2) * NCOL * 4), "s"(outp) : "memory"); \
      } while (0)
      while (ok && t < U_END) {
        if (((t - U_C) >> 5) >= ready) { POLLC(t); if (!ok) break; }
        RDESC(t, toff, ooff);
        GLOAD2(ra0, ra1, rsp, toff + lo16, 0);
        GLOAD2(la0, la1, lso, toff + lo16, 0);
        DRAW();
        asm volatile("s_waitcnt vmcnt(0)" ::: "memory");
        for (;;) {
          GLOAD2(rb0, rb1, rsp, toff + lo16, 2048);
          GLOAD2(lb0, lb1, lso, toff + lo16, 2048);
          asm volatile("s_waitcnt vmcnt(6)" : "+v"(ra0), "+v"(ra1), "+v"(la0), "+v"(la1), "+v"(tv) :: "memory");
          RFIN(ra0, ra1, la0, la1, 0);
          t = (int)(__builtin_amdgcn_readfirstlane(tv) - qbase);
          const bool more = (t < U_END) && (((t - U_C) >> 5) < ready);
          if (more) {
            RDESC(t, ntoff, nooff);
            GLOAD2(ra0, ra1, rsp, ntoff + lo16, 0);
            GLOAD2(la0, la1, lso, ntoff + lo16, 0);
            DRAW();
            asm volatile("s_waitcnt vmcnt(7)" ::: "memory");
          } else {
            asm volatile("s_waitcnt vmcnt(2)" ::: "memory");
          }
          asm volatile("" : "+v"(rb0), "+v"(rb1), "+v"(lb0), "+v"(lb1) :: "memory");
          RFIN(rb0, rb1, lb0, lb1, 1);
          if (!more) break;
          toff = ntoff; ooff = nooff;
        }
      }
      while (t < U_END) {
        RDESC(t, toff, ooff);
        float nv = __builtin_nanf("");
        f4 q = f4{nv, nv, nv, nv};
        #pragma unroll
        for (int k = 0; k < 4; ++k) *(f4*)(outp + ooff + olane + k * 2 * NCOL * 4) = q;
        DRAW();
        asm volatile("s_waitcnt vmcnt(0)" : "+v"(tv) :: "memory");
        t = (int)(__builtin_amdgcn_readfirstlane(tv) - qbase);
      }
    }
    }
    STAMP(5);
  }
}

}

extern "C" void kernel_launch(int rank, int nranks, ncclComm_t comm,
                              hipStream_t stream,
                              void* const* d_in, const int* in_sizes, int n_in,
                              void* d_out, int out_size,
                              const int* in_dtypes, int out_dtype,
                              void* const* d_out_all,
                              void* const* d_ws_all, size_t ws_bytes) {
  (void)comm; (void)in_sizes; (void)n_in; (void)out_size; (void)in_dtypes; (void)out_dtype; (void)d_out_all;
  if (nranks != NRANK || ws_bytes < WS_NEED) {
    fprintf(stderr, "unsupported configuration: nranks %d ws %zu\n", nranks, ws_bytes);
    return;
  }
  HIP_CHECK(hipFuncSetAttribute((const void*)fused_gemm_rs, hipFuncAttributeMaxDynamicSharedMemorySize, LDS_BYTES));
  Params P;
  P.x = (const float*)d_in[0];
  P.w = (const float*)d_in[1];
  P.out = (float*)d_out;
  for (int e = 0; e < NRANK; ++e) P.wsx[e] = (char*)d_ws_all[rank ^ e];
  P.rank = rank;
  fused_gemm_rs<<<NWG, NTHR, LDS_BYTES, stream>>>(P);
  HIP_CHECK(hipGetLastError());
}
```

```cpp
#include <hip/hip_runtime.h>
#include <rccl/rccl.h>
#include <cstdio>
#include <cstdint>

#define HIP_CHECK(expr) do { hipError_t _e = (expr); if (_e != hipSuccess) { \
    fprintf(stderr, "HIP error %s at %s:%d\n", hipGetErrorString(_e), __FILE__, __LINE__); \
    return; } } while (0)

namespace {

typedef _Float16 h8 __attribute__((ext_vector_type(8)));
typedef float f4 __attribute__((ext_vector_type(4)));
typedef unsigned u4 __attribute__((ext_vector_type(4)));

constexpr int NRANK = 8;
constexpr int NCOL = 16384;
constexpr int KS = 512;
constexpr int BN = 128;
constexpr int NCT = NCOL / BN;
constexpr int NWG = 256;
constexpr int NGEMM = 224;
constexpr int NTHR = 512;
#define PW_PER_WG 5
constexpr int PWAVES = 4 * PW_PER_WG;

constexpr size_t MiB = 1024 * 1024;
constexpr unsigned OFF_EPOCH = 0;
constexpr unsigned OFF_DONE = 512;
constexpr unsigned OFF_TMO = 128;
constexpr unsigned OFF_XRDY = 1024;
constexpr unsigned OFF_WRDY = 2048;
constexpr unsigned OFF_Q = 3072;
constexpr unsigned OFF_CX = 4096;
constexpr unsigned OFF_CW = 6144;
constexpr unsigned OFF_XCNT = 8192;
constexpr unsigned OFF_WCNT = 32768;
constexpr unsigned OFF_FLC = 65536;
constexpr unsigned OFF_FLO = 69632;
constexpr unsigned OFF_DBG = 131072;
constexpr size_t OFF_WC = 1 * MiB;
constexpr size_t OFF_XC = OFF_WC + 32 * MiB;
constexpr size_t OFF_WP = OFF_XC + 8 * MiB;
constexpr size_t OFF_XP = OFF_WP + 16 * MiB;
constexpr size_t OFF_SO = OFF_XP + 32 * MiB;
constexpr size_t OFF_SP = OFF_SO + 32 * MiB;
constexpr size_t WS_NEED = OFF_SP + 64 * MiB;

constexpr unsigned STAGE_B = 49152;
constexpr unsigned LDS_BC = 3 * STAGE_B;
constexpr unsigned LDS_BYTES = LDS_BC + 64;

constexpr unsigned long long DEADLINE_TICKS = 50000000ull;

#define NXA 2
constexpr int NXB = 12 - NXA;
constexpr unsigned long long XSEQ = 0xedc6b5a49321ull;
constexpr int U_W = NXA * 64;
constexpr int U_X2 = U_W + 512;
constexpr int U_XO = U_X2 + NXB * 64;
constexpr int U_C = U_XO + 128;
constexpr int U_END = U_C + 1024;

struct Params {
  const float* x;
  const float* w;
  float* out;
  char* wsx[NRANK];
  int rank;
};
__device__ __forceinline__ char* peer_ws(const Params& P, int e) {
  char* p = P.wsx[1];
  p = (e == 2) ? P.wsx[2] : p; p = (e == 3) ? P.wsx[3] : p; p = (e == 4) ? P.wsx[4] : p;
  p = (e == 5) ? P.wsx[5] : p; p = (e == 6) ? P.wsx[6] : p; p = (e == 7) ? P.wsx[7] : p;
  return p;
}

__device__ __forceinline__ unsigned ld_agent(const char* p) {
  return __hip_atomic_load((const unsigned*)p, __ATOMIC_RELAXED, __HIP_MEMORY_SCOPE_AGENT);
}
__device__ __forceinline__ unsigned ld_sys(const char* p) {
  return __hip_atomic_load((const unsigned*)p, __ATOMIC_RELAXED, __HIP_MEMORY_SCOPE_SYSTEM);
}
__device__ __forceinline__ void st_sys(char* p, unsigned v) {
  __hip_atomic_store((unsigned*)p, v, __ATOMIC_RELAXED, __HIP_MEMORY_SCOPE_SYSTEM);
}
#define VMCNT0() asm volatile("s_waitcnt vmcnt(0)" ::: "memory")

template <int SLEEP>
__device__ __forceinline__ bool wave_wait(const char* word, unsigned target, char* tmo, unsigned call) {
  unsigned v = ld_sys(word);
  if (__all((int)(v - target) >= 0)) return true;
  unsigned long long t0 = __builtin_amdgcn_s_memrealtime();
  for (;;) {
    __builtin_amdgcn_s_sleep(SLEEP);
    v = ld_sys(word);
    if (__all((int)(v - target) >= 0)) return true;
    if (ld_agent(tmo) == call || __builtin_amdgcn_s_memrealtime() - t0 > DEADLINE_TICKS) {
      __hip_atomic_store((unsigned*)tmo, call, __ATOMIC_RELAXED, __HIP_MEMORY_SCOPE_AGENT);
      return false;
    }
  }
}
#define WCAUX 17
#define GSLEEP 32

__device__ __forceinline__ unsigned pack2(float a, float b) {
  typedef _Float16 h2 __attribute__((ext_vector_type(2)));
  h2 v = {(_Float16)a, (_Float16)b};
  return __builtin_bit_cast(unsigned, v);
}

#define GLOAD2(D0, D1, BASE, VOFF, IMM) \
  asm volatile("global_load_dwordx4 %0, %2, %3 offset:%4 sc0 sc1\n\tglobal_load_dwordx4 %1, %2, %3 offset:%5 sc0 sc1" \
               : "=&v"(D0), "=&v"(D1) : "v"(VOFF), "s"(BASE), "i"(IMM), "i"(IMM + 1024) : "memory")
#define GSTORE2(D0, D1, BASE, VOFF, IMM) \
  asm volatile("global_store_dwordx4 %2, %0, %3 offset:%4 sc0 sc1\n\tglobal_store_dwordx4 %2, %1, %3 offset:%5 sc0 sc1\n\ts_nop 1" \
               :: "v"(D0), "v"(D1), "v"(VOFF), "s"(BASE), "i"(IMM), "i"(IMM + 1024) : "memory")
#define DRAW() do { if (lane == 0) asm volatile("global_atomic_add %0, %1, %2, %3 sc0" : "+v"(tv) : "v"(qoff), "v"(1u), "s"(lws) : "memory"); } while (0)
#define SIGNAL(OFFS) do { if (lane == 0) asm volatile("global_atomic_add %0, %1, %2" :: "v"(OFFS), "v"(1u), "s"(lws) : "memory"); } while (0)

#define STAMP(slot) do {} while (0)

template <int NI>
__device__ __forceinline__ void gemm_tile(f4 (&acc)[4][4], char* smem, char* lws, char* tmo, unsigned call, int nsh,
                                          const char* a_loc, const char* b_loc, int e_first, int xsel, int ct, int cs,
                                          int wave, int lane) {
  const int wm = wave >> 1, wn = wave & 1;
  #pragma unroll
  for (int mi = 0; mi < 4; ++mi)
    #pragma unroll
    for (int ni = 0; ni < NI; ++ni) acc[mi][ni] = f4{0.f, 0.f, 0.f, 0.f};
  const int nt = nsh * 8;
  const int aoff = wave * 4096 + lane * 16;
  const int boff = (NI == 4) ? wave * 2048 + lane * 16 : (wave >> 2) * 8192 + cs * 4096 + (wave & 3) * 1024 + lane * 16;
  const char* ap = a_loc + aoff;
  const char* bp = b_loc + boff;
#define ISSUE_STAGE(BUF) do { \
      char* dst_ = smem + (BUF) * STAGE_B; \
      _Pragma("unroll") for (int i = 0; i < 4; ++i) \
        __builtin_amdgcn_global_load_lds((const unsigned*)(ap + i * 1024), (unsigned*)(dst_ + wave * 4096 + i * 1024), 16, 0, 16); \
      _Pragma("unroll") for (int i = 0; i < NI / 2; ++i) \
        __builtin_amdgcn_global_load_lds((const unsigned*)(bp + i * 1024), (unsigned*)(dst_ + 32768 + wave * (512 * NI) + i * 1024), 16, 0, 16); \
      ap += 32768; bp += 16384; } while (0)
#define READ_FRAGS(AF, BF, BUF, KB) do { \
      const char* sa_ = smem + (BUF) * STAGE_B + (KB) * 16384 + wm * 4096 + lane * 16; \
      const char* sb_ = smem + (BUF) * STAGE_B + 32768 + (KB) * (2048 * NI) + wn * (1024 * NI) + lane * 16; \
      _Pragma("unroll") for (int i = 0; i < 4; ++i) { if (i < NI) BF[i] = *(const h8*)(sb_ + i * 1024); AF[i] = *(const h8*)(sa_ + i * 1024); } } while (0)
#define MMA(AF, BF) do { \
      __builtin_amdgcn_s_setprio(1); \
      _Pragma("unroll") for (int mi = 0; mi < 4; ++mi) \
        _Pragma("unroll") for (int ni = 0; ni < NI; ++ni) \
          acc[mi][ni] = __builtin_amdgcn_mfma_f32_16x16x32_f16(BF[ni], AF[mi], acc[mi][ni], 0, 0, 0); \
      __builtin_amdgcn_s_setprio(0); } while (0)
  h8 a0[4], b0[NI], a1[4], b1[NI];
  ISSUE_STAGE(0); ISSUE_STAGE(1); ISSUE_STAGE(2);
  if (NI == 4) asm volatile("s_waitcnt vmcnt(12)" ::: "memory");
  else asm volatile("s_waitcnt vmcnt(10)" ::: "memory");
  __builtin_amdgcn_s_barrier();
  READ_FRAGS(a0, b0, 0, 0);
  int buf = 0;
  for (int t = 0; t < nt; ++t) {
    READ_FRAGS(a1, b1, buf, 1);
    __builtin_amdgcn_sched_barrier(0);
    MMA(a0, b0);
    __builtin_amdgcn_sched_barrier(0);
    const int nbuf = (buf == 2) ? 0 : buf + 1;
    if (t + 1 < nt) {
      const int ti = t + 3;
      const bool next_shard = ti < nt && (ti & 7) == 0;
      int e = e_first - (ti >> 3); e += (e < 0) ? 7 : 0; e += 1;
      if (next_shard && wave == 0) {
        const char* word = (lane & 1) ? (lws + OFF_WCNT + (e * 16 + ct) * 128) : (lws + OFF_XCNT + (e * 16 + xsel) * 128);
        unsigned target = (lane & 1) ? call * 32u : call * 64u;
        (void)wave_wait<GSLEEP>(word, target, tmo, call);
      }
      if (t + 2 < nt) {
        if (NI == 4) asm volatile("s_waitcnt vmcnt(6) lgkmcnt(0)" ::: "memory");
        else asm volatile("s_waitcnt vmcnt(5) lgkmcnt(0)" ::: "memory");
      } else asm volatile("s_waitcnt vmcnt(0) lgkmcnt(0)" ::: "memory");
      __builtin_amdgcn_s_barrier();
      if (ti < nt) {
        if (next_shard) {
          ap = lws + OFF_XP + (size_t)(e * 16 + xsel) * 262144 + aoff;
          bp = lws + OFF_WP + (size_t)(e * 16 + ct) * 131072 + boff;
        }
        ISSUE_STAGE(buf);
      }
      READ_FRAGS(a0, b0, nbuf, 0);
    }
    __builtin_amdgcn_sched_barrier(0);
    MMA(a1, b1);
    __builtin_amdgcn_sched_barrier(0);
    buf = nbuf;
  }
}

__global__ void __launch_bounds__(NTHR, 2) fused_gemm_rs(Params P) {
  extern __shared__ __attribute__((aligned(16))) char smem[];
  const int tid = threadIdx.x, lane = tid & 63;
  const int wave = __builtin_amdgcn_readfirstlane(tid >> 6);
  const int bid = blockIdx.x;
  const int rank = P.rank;
  char* const lws = P.wsx[0];
  char* const tmo = lws + OFF_TMO;

  STAMP(0);
  const unsigned epoch = ld_agent(lws + OFF_EPOCH);
  f4 xv[2][2];
  {
    #pragma unroll
    for (int i = 0; i < 2; ++i) {
      int F = (bid * 8 + wave) * 2 + i;
      int rh = F >> 8, kb = (F >> 4) & 15, rb = F & 15;
      const float* src = P.x + (size_t)(rh * 256 + rb * 16 + (lane & 15)) * KS + kb * 32 + (lane >> 4) * 8;
      xv[i][0] = *(const f4*)src;
      xv[i][1] = *(const f4*)(src + 4);
    }
  }
  const unsigned call = __builtin_amdgcn_readfirstlane(epoch) + 1;
  const unsigned par = call & 1;

  auto Lr = __builtin_amdgcn_make_buffer_rsrc((void*)lws, 0, (int)WS_NEED, 0x00020000);

  {
    u4 o[2];
    #pragma unroll
    for (int i = 0; i < 2; ++i) {
      o[i][0] = pack2(xv[i][0][0], xv[i][0][1]);
      o[i][1] = pack2(xv[i][0][2], xv[i][0][3]);
      o[i][2] = pack2(xv[i][1][0], xv[i][1][1]);
      o[i][3] = pack2(xv[i][1][2], xv[i][1][3]);
    }
    const unsigned lofs = (unsigned)(OFF_XC + par * 4 * MiB) + (bid * 8 + wave) * 2048 + lane * 16;
    GSTORE2(o[0], o[1], lws, lofs, 0);
    VMCNT0();
    __builtin_amdgcn_s_barrier();
    if (wave == 0) SIGNAL(OFF_CX + (bid >> 4) * 128);
    if (bid == NWG - 1 && wave == 0) {
      if (wave_wait<1>(lws + OFF_CX + (lane & 15) * 128, call * 16u, tmo, call)) {
        if (lane >= 1 && lane < 8) st_sys(peer_ws(P, lane) + OFF_XRDY + lane * 128, call);
        if (wave_wait<4>(lws + OFF_CW + (lane & 7) * 128, call * (unsigned)(NGEMM / 8), tmo, call)) {
          if (lane < 8) st_sys((lane ? peer_ws(P, lane) : lws) + OFF_WRDY + lane * 128, call);
        }
      }
    }
  }
  STAMP(1);

  if (bid < NGEMM) {
    {
      const int T0 = bid * 8 + wave;
      const int T2 = 2 * NGEMM * 8 + wave * NGEMM + bid;
      const bool third = T2 < 4096;
      float v[3][4][8];
      #pragma unroll
      for (int r = 0; r < 3; ++r) {
        if (r == 2 && !third) break;
        const int T = (r == 2) ? T2 : T0 + r * NGEMM * 8;
        const float* src = P.w + (size_t)((T >> 6) * 8) * NCOL + (T & 63) * 256 + lane;
        #pragma unroll
        for (int j = 0; j < 8; ++j)
          #pragma unroll
          for (int i = 0; i < 4; ++i) v[r][i][j] = src[(size_t)j * NCOL + i * 64];
      }
      #pragma unroll
      for (int r = 0; r < 3; ++r) {
        if (r == 2 && !third) break;
        const int T = (r == 2) ? T2 : T0 + r * NGEMM * 8;
        const int cg = T & 63, ko = T >> 6;
        int dst = (int)(OFF_WC + par * 16 * MiB) + cg * 2 * 131072 + (ko >> 2) * 8192 + (lane >> 4) * 1024 + ((ko & 3) * 16 + (lane & 15)) * 16;
        #pragma unroll
        for (int i = 0; i < 4; ++i) {
          u4 o;
          o[0] = pack2(v[r][i][0], v[r][i][1]);
          o[1] = pack2(v[r][i][2], v[r][i][3]);
          o[2] = pack2(v[r][i][4], v[r][i][5]);
          o[3] = pack2(v[r][i][6], v[r][i][7]);
          __builtin_amdgcn_raw_buffer_store_b128(o, Lr, dst + (i >> 1) * 131072 + (i & 1) * 4096, 0, WCAUX);
        }
      }
      VMCNT0();
      __syncthreads();
      if (wave == 0) SIGNAL(OFF_CW + (bid & 7) * 128);
    }
    STAMP(2);
    {
      if (wave == 0) (void)wave_wait<GSLEEP>(lws + OFF_WRDY, call, tmo, call);
      __syncthreads();
    }
    STAMP(3);

    const int xcd = bid & 7, slot = bid >> 3;
    const int ct = 4 * (xcd >> 1) + (slot & 3);
    const int h = xcd & 1, dq = (slot >> 2) + 1;
    const int wm = wave >> 1, wn = wave & 1;
    const char* const xc = lws + OFF_XC + par * 4 * MiB;
    const char* const wc = lws + OFF_WC + par * 16 * MiB;
    f4 acc[4][4];

    for (int job = 0; job < 2; ++job) {
      int nsh, e_first, xsel;
      const char *a_loc, *b_loc;
      if (job == 0) {
        nsh = 1; e_first = 0; xsel = 0;
        a_loc = xc + (size_t)(rank * 2 + h) * 262144;
        b_loc = wc + (size_t)((rank ^ dq) * 16 + ct) * 131072;
      } else {
        nsh = 7; e_first = dq - 1; xsel = dq * 2 + h;
        a_loc = xc + (size_t)((rank ^ dq) * 2 + h) * 262144;
        b_loc = wc + (size_t)(rank * 16 + ct) * 131072;
      }
      gemm_tile<4>(acc, smem, lws, tmo, call, nsh, a_loc, b_loc, e_first, xsel, ct, 0, wave, lane);
      int dst = (job == 0) ? (int)OFF_SO : (int)(OFF_SP + par * 32 * MiB);
      dst += ((dq * 2 + h) * 16 + ct) * 131072 + (wm * 64 + (lane & 15)) * 512 + wn * 256 + (lane >> 4) * 16;
      #pragma unroll
      for (int mi = 0; mi < 4; ++mi)
        #pragma unroll
        for (int ni = 0; ni < 4; ++ni)
          __builtin_amdgcn_raw_buffer_store_b128(__builtin_bit_cast(u4, acc[mi][ni]), Lr, dst + mi * 8192 + ni * 64, 0, 17);
      VMCNT0();
      __syncthreads();
      if (tid == 0) {
        int fu = (dq * 32 + h * 16 + ct) * 4;
        if (job == 0) st_sys(lws + OFF_FLO + fu, call);
        else st_sys(peer_ws(P, dq) + OFF_FLC + fu, call);
      }
      STAMP(4 + job);
    }
    if (h == 0 && dq <= 4) {
      const int ho = (dq - 1) >> 1, cs = (dq - 1) & 1;
      gemm_tile<2>(acc, smem, lws, tmo, call, 8, xc + (size_t)(rank * 2 + ho) * 262144, wc + (size_t)(rank * 16 + ct) * 131072,
                   7, ho, ct, cs, wave, lane);
      const bool bad = ld_agent(tmo) == call;
      float* o = P.out + (size_t)(ho * 256 + wm * 64 + (lane & 15)) * NCOL + (rank * 16 + ct) * BN + cs * 64 + wn * 32 + (lane >> 4) * 4;
      #pragma unroll
      for (int mi = 0; mi < 4; ++mi)
        #pragma unroll
        for (int ni = 0; ni < 2; ++ni) {
          f4 v = acc[mi][ni];
          #pragma unroll
          for (int k = 0; k < 4; ++k) v[k] = bad ? __builtin_nanf("") : fmaxf(v[k], 0.f);
          *(f4*)(o + (size_t)mi * 16 * NCOL + ni * 16) = v;
        }
      STAMP(6);
    }
  } else if (bid < NGEMM + 28) {
    const int pw = bid - NGEMM;
    const int e = (pw % 7) + 1;
    char* const rws = peer_ws(P, e);
    const unsigned qbase = (call - 1) * (unsigned)(U_END + PWAVES);
    const unsigned qoff = OFF_Q + e * 128;
    const unsigned lo16 = lane * 16;
    if (wave < PW_PER_WG) {
    unsigned tv = 0;
    DRAW();
    bool ok = wave_wait<2>(lws + OFF_XRDY + e * 128, call, tmo, call);
    STAMP(2);
    asm volatile("s_waitcnt vmcnt(0)" : "+v"(tv) :: "memory");
    int t = (int)(__builtin_amdgcn_readfirstlane(tv) - qbase);
    u4 ra0, ra1, rb0, rb1;

#define CDESC(t, SO, LO, CN) do { \
      if ((t) >= U_W && (t) < U_X2) { \
        if (!wseen) { wseen = true; STAMP(3); ok = ok && wave_wait<2>(lws + OFF_WRDY + e * 128, call, tmo, call); } \
        int ct_ = ((t) - U_W) >> 5; \
        SO = (unsigned)(OFF_WC + par * 16 * MiB) + (rank * 16 + ct_) * 131072 + ((t) & 31) * 4096; \
        LO = (unsigned)OFF_WP + (e * 16 + ct_) * 131072 + ((t) & 31) * 4096; \
        CN = OFF_WCNT + (e * 16 + ct_) * 128; \
      } else { \
        int dq_, h_; \
        if ((t) >= U_XO) { dq_ = 0; h_ = ((t) - U_XO) >> 6; } \
        else { \
          int hk_ = (int)(XSEQ >> (4 * (((t) < U_W) ? ((t) >> 6) : NXA + (((t) - U_X2) >> 6)))) & 15; \
          h_ = hk_ >> 3; \
          dq_ = e - 1 + (hk_ & 7); dq_ -= (dq_ >= 7) ? 7 : 0; dq_ += 1; \
        } \
        SO = (unsigned)(OFF_XC + par * 4 * MiB) + ((rank ^ dq_) * 2 + h_) * 262144 + ((t) & 63) * 4096; \
        LO = (unsigned)OFF_XP + ((e * 8 + dq_) * 2 + h_) * 262144 + ((t) & 63) * 4096; \
        CN = OFF_XCNT + ((e * 8 + dq_) * 2 + h_) * 128; \
      } } while (0)
    if (ok && t < U_C) {
      bool wseen = false;
      unsigned pending = 0;
      unsigned soff, loff, cnt, nsoff = 0, nloff = 0, ncnt = 0;
      CDESC(t, soff, loff, cnt);
      GLOAD2(ra0, ra1, rws, soff + lo16, 0);
      DRAW();
      asm volatile("s_waitcnt vmcnt(0)" ::: "memory");
      for (;;) {
        GLOAD2(rb0, rb1, rws, soff + lo16, 2048);
        asm volatile("s_waitcnt vmcnt(4)" : "+v"(tv) :: "memory");
        GSTORE2(ra0, ra1, lws, loff + lo16, 0);
        t = (int)(__builtin_amdgcn_readfirstlane(tv) - qbase);
        const bool more = t < U_C;
        if (more) {
          CDESC(t, nsoff, nloff, ncnt);
          GLOAD2(ra0, ra1, rws, nsoff + lo16, 0);
          DRAW();
          asm volatile("s_waitcnt vmcnt(5)" ::: "memory");
        } else {
          asm volatile("s_waitcnt vmcnt(2)" ::: "memory");
        }
        if (pending) SIGNAL(pending);
        pending = cnt;
        GSTORE2(rb0, rb1, lws, loff + lo16, 2048);
        if (!more || !ok) break;
        soff = nsoff; loff = nloff; cnt = ncnt;
      }
      asm volatile("s_waitcnt vmcnt(0)" : "+v"(tv) :: "memory");
      if (ok) SIGNAL(pending);
      else {
        while (t < U_C) { t = (int)(__builtin_amdgcn_readfirstlane(tv) - qbase); if (t < U_C) { DRAW(); asm volatile("s_waitcnt vmcnt(0)" : "+v"(tv) :: "memory"); } }
      }
    } else if (!ok) {
      while (t < U_C) { DRAW(); asm volatile("s_waitcnt vmcnt(0)" : "+v"(tv) :: "memory"); t = (int)(__builtin_amdgcn_readfirstlane(tv) - qbase); }
    }
    STAMP(4);
    {
      int ready = 0;
      const unsigned olane = ((lane >> 5) * NCOL + (lane & 31) * 4) * 4;
      u4 la0, la1, lb0, lb1;
      unsigned toff = 0, ooff = 0, ntoff = 0, nooff = 0;
      const char* const rsp = rws + OFF_SP + par * 32 * MiB + (size_t)e * 32 * 131072;
      const char* const lso = lws + OFF_SO + (size_t)e * 32 * 131072;
      char* const outp = (char*)(P.out + (rank ^ e) * 2048);
#define RDESC(t, TO, OO) do { \
        const int u_ = ((t) - U_C) >> 5, wv_ = (t) & 31; \
        TO = ((t) - U_C) * 4096; \
        OO = (((u_ >> 4) * 256 + wv_ * 8) * NCOL + (u_ & 15) * BN) * 4; \
      } while (0)
#define POLLC(t) do { \
      const int u_ = ((t) - U_C) >> 5; \
      unsigned long long t0_ = __builtin_amdgcn_s_memrealtime(); \
      for (;;) { \
        int uu_ = u_ + lane; if (uu_ > 31) uu_ = 31; \
        unsigned a_ = ld_sys(lws + OFF_FLC + (e * 32 + uu_) * 4); \
        unsigned b_ = ld_sys(lws + OFF_FLO + (e * 32 + uu_) * 4); \
        bool good_ = ((int)(a_ - call) >= 0) && ((int)(b_ - call) >= 0) && (u_ + lane < 32); \
        unsigned long long m_ = __ballot(good_); \
        int n_ = __builtin_ctzll(~m_); \
        if (n_ > 0) { ready = u_ + n_; break; } \
        if (ld_agent(tmo) == call || __builtin_amdgcn_s_memrealtime() - t0_ > DEADLINE_TICKS) { \
          __hip_atomic_store((unsigned*)tmo, call, __ATOMIC_RELAXED, __HIP_MEMORY_SCOPE_AGENT); \
          ok = false; break; \
        } \
        __builtin_amdgcn_s_sleep(2); \
      } \
      asm volatile("s_waitcnt vmcnt(0)" ::: "memory"); } while (0)
#define RFIN(R0, R1, L0, L1, IMM) do { \
        f4 r0_ = __builtin_bit_cast(f4, R0) + __builtin_bit_cast(f4, L0); \
        f4 r1_ = __builtin_bit_cast(f4, R1) + __builtin_bit_cast(f4, L1); \
        _Pragma("unroll") for (int k_ = 0; k_ < 4; ++k_) { r0_[k_] = fmaxf(r0_[k_], 0.f); r1_[k_] = fmaxf(r1_[k_], 0.f); } \
        asm volatile("global_store_dwordx4 %0, %1, %4\n\tglobal_store_dwordx4 %3, %2, %4\n\ts_nop 1" \
                     :: "v"(ooff + olane + (IMM) * 4 * NCOL * 4), "v"(r0_), "v"(r1_), "v"(ooff + olane + ((IMM) * 4 + 2) * NCOL * 4), "s"(outp) : "memory"); \
      } while (0)
      while (ok && t < U_END) {
        if (((t - U_C) >> 5) >= ready) { POLLC(t); if (!ok) break; }
        RDESC(t, toff, ooff);
        GLOAD2(ra0, ra1, rsp, toff + lo16, 0);
        GLOAD2(la0, la1, lso, toff + lo16, 0);
        DRAW();
        asm volatile("s_waitcnt vmcnt(0)" ::: "memory");
        for (;;) {
          GLOAD2(rb0, rb1, rsp, toff + lo16, 2048);
          GLOAD2(lb0, lb1, lso, toff + lo16, 2048);
          asm volatile("s_waitcnt vmcnt(6)" : "+v"(ra0), "+v"(ra1), "+v"(la0), "+v"(la1), "+v"(tv) :: "memory");
          RFIN(ra0, ra1, la0, la1, 0);
          t = (int)(__builtin_amdgcn_readfirstlane(tv) - qbase);
          const bool more = (t < U_END) && (((t - U_C) >> 5) < ready);
          if (more) {
            RDESC(t, ntoff, nooff);
            GLOAD2(ra0, ra1, rsp, ntoff + lo16, 0);
            GLOAD2(la0, la1, lso, ntoff + lo16, 0);
            DRAW();
            asm volatile("s_waitcnt vmcnt(7)" ::: "memory");
          } else {
            asm volatile("s_waitcnt vmcnt(2)" ::: "memory");
          }
          asm volatile("" : "+v"(rb0), "+v"(rb1), "+v"(lb0), "+v"(lb1) :: "memory");
          RFIN(rb0, rb1, lb0, lb1, 1);
          if (!more) break;
          toff = ntoff; ooff = nooff;
        }
      }
      while (t < U_END) {
        RDESC(t, toff, ooff);
        float nv = __builtin_nanf("");
        f4 q = f4{nv, nv, nv, nv};
        #pragma unroll
        for (int k = 0; k < 4; ++k) *(f4*)(outp + ooff + olane + k * 2 * NCOL * 4) = q;
        DRAW();
        asm volatile("s_waitcnt vmcnt(0)" : "+v"(tv) :: "memory");
        t = (int)(__builtin_amdgcn_readfirstlane(tv) - qbase);
      }
    }
    }
    STAMP(5);
  }
  __syncthreads();
  if (tid == 0) {
    unsigned old = __hip_atomic_fetch_add((unsigned*)(lws + OFF_DONE), 1u, __ATOMIC_RELAXED, __HIP_MEMORY_SCOPE_AGENT);
    if (old == call * NWG - 1) __hip_atomic_store((unsigned*)(lws + OFF_EPOCH), call, __ATOMIC_RELAXED, __HIP_MEMORY_SCOPE_AGENT);
  }
}

}

extern "C" void kernel_launch(int rank, int nranks, ncclComm_t comm,
                              hipStream_t stream,
                              void* const* d_in, const int* in_sizes, int n_in,
                              void* d_out, int out_size,
                              const int* in_dtypes, int out_dtype,
                              void* const* d_out_all,
                              void* const* d_ws_all, size_t ws_bytes) {
  (void)comm; (void)in_sizes; (void)n_in; (void)out_size; (void)in_dtypes; (void)out_dtype; (void)d_out_all;
  if (nranks != NRANK || ws_bytes < WS_NEED) {
    fprintf(stderr, "unsupported configuration: nranks %d ws %zu\n", nranks, ws_bytes);
    return;
  }
  HIP_CHECK(hipFuncSetAttribute((const void*)fused_gemm_rs, hipFuncAttributeMaxDynamicSharedMemorySize, LDS_BYTES));
  Params P;
  P.x = (const float*)d_in[0];
  P.w = (const float*)d_in[1];
  P.out = (float*)d_out;
  for (int e = 0; e < NRANK; ++e) P.wsx[e] = (char*)d_ws_all[rank ^ e];
  P.rank = rank;
  fused_gemm_rs<<<NWG, NTHR, LDS_BYTES, stream>>>(P);
  HIP_CHECK(hipGetLastError());
}
```

```cpp
#include <hip/hip_runtime.h>
#include <rccl/rccl.h>
#include <cstdio>
#include <cstdint>

#define HIP_CHECK(expr) do { hipError_t _e = (expr); if (_e != hipSuccess) { \
    fprintf(stderr, "HIP error %s at %s:%d\n", hipGetErrorString(_e), __FILE__, __LINE__); \
    return; } } while (0)

namespace {

typedef _Float16 h8 __attribute__((ext_vector_type(8)));
typedef float f4 __attribute__((ext_vector_type(4)));
typedef unsigned u4 __attribute__((ext_vector_type(4)));

constexpr int NRANK = 8;
constexpr int NCOL = 16384;
constexpr int KS = 512;
constexpr int BN = 128;
constexpr int NCT = NCOL / BN;
constexpr int NWG = 256;
constexpr int NGEMM = 224;
constexpr int NTHR = 512;
#define PW_PER_WG 5
constexpr int PWAVES = 4 * PW_PER_WG;

constexpr size_t MiB = 1024 * 1024;
constexpr unsigned OFF_EPOCH = 0;
constexpr unsigned OFF_TMO = 128;
constexpr unsigned OFF_XRDY = 1024;
constexpr unsigned OFF_WRDY = 2048;
constexpr unsigned OFF_Q = 3072;
constexpr unsigned OFF_CX = 4096;
constexpr unsigned OFF_CW = 6144;
constexpr unsigned OFF_XCNT = 8192;
constexpr unsigned OFF_WCNT = 32768;
constexpr unsigned OFF_Q2 = 7168;
constexpr unsigned OFF_FLC = 65536;
constexpr unsigned OFF_DBG = 131072;
constexpr size_t OFF_WC = 1 * MiB;
constexpr size_t OFF_XC = OFF_WC + 32 * MiB;
constexpr size_t OFF_WP = OFF_XC + 8 * MiB;
constexpr size_t OFF_XP = OFF_WP + 16 * MiB;
constexpr unsigned CHUNK_B = 8448;
constexpr size_t SP_BYTES = 34 * MiB;
constexpr size_t OFF_SP = OFF_XP + 32 * MiB;
constexpr size_t WS_NEED = OFF_SP + 2 * SP_BYTES;

constexpr unsigned STAGE_B = 49152;
constexpr unsigned LDS_BC = 3 * STAGE_B;
constexpr unsigned LDS_BYTES = LDS_BC + 64;

constexpr unsigned long long DEADLINE_TICKS = 50000000ull;

constexpr int NXA = 2, NXB = 14;
#define OWNPOS 12
constexpr unsigned long long XSEQ = OWNPOS == 14 ? 0xf7edcb65a4938210ull : OWNPOS == 12 ? 0xedf7cb65a4938210ull : OWNPOS == 11 ? 0xedcf7b65a4938210ull : 0xedcbf765a4938210ull;
constexpr int U_W = NXA * 64;
constexpr int U_X2 = U_W + 512;
constexpr int U_C = U_X2 + NXB * 64;
constexpr int NCHUNK = 512;

struct Params {
  const float* x;
  const float* w;
  float* out;
  char* wsx[NRANK];
  int rank;
};
__device__ __forceinline__ char* peer_ws(const Params& P, int e) {
  char* p = P.wsx[1];
  p = (e == 2) ? P.wsx[2] : p; p = (e == 3) ? P.wsx[3] : p; p = (e == 4) ? P.wsx[4] : p;
  p = (e == 5) ? P.wsx[5] : p; p = (e == 6) ? P.wsx[6] : p; p = (e == 7) ? P.wsx[7] : p;
  return p;
}

__device__ __forceinline__ unsigned ld_agent(const char* p) {
  return __hip_atomic_load((const unsigned*)p, __ATOMIC_RELAXED, __HIP_MEMORY_SCOPE_AGENT);
}
__device__ __forceinline__ unsigned ld_sys(const char* p) {
  return __hip_atomic_load((const unsigned*)p, __ATOMIC_RELAXED, __HIP_MEMORY_SCOPE_SYSTEM);
}
__device__ __forceinline__ void st_sys(char* p, unsigned v) {
  __hip_atomic_store((unsigned*)p, v, __ATOMIC_RELAXED, __HIP_MEMORY_SCOPE_SYSTEM);
}
#define VMCNT0() asm volatile("s_waitcnt vmcnt(0)" ::: "memory")

template <int SLEEP>
__device__ __forceinline__ bool wave_wait(const char* word, unsigned target, char* tmo, unsigned call) {
  unsigned v = ld_sys(word);
  if (__all((int)(v - target) >= 0)) return true;
  unsigned long long t0 = __builtin_amdgcn_s_memrealtime();
  for (;;) {
    __builtin_amdgcn_s_sleep(SLEEP);
    v = ld_sys(word);
    if (__all((int)(v - target) >= 0)) return true;
    if (ld_agent(tmo) == call || __builtin_amdgcn_s_memrealtime() - t0 > DEADLINE_TICKS) {
      __hip_atomic_store((unsigned*)tmo, call, __ATOMIC_RELAXED, __HIP_MEMORY_SCOPE_AGENT);
      return false;
    }
  }
}
#define WCAUX 17
#define OUTPOL ""
#define UW_PER_WG 3
#define UHELP 4
#define OUTMODE 2
#define OUT_STORE(P_, V_) asm volatile("global_store_dwordx4 %0, %1, off sc0 sc1\n\ts_nop 1" :: "v"(P_), "v"(V_) : "memory")
#define GSLEEP 32

__device__ __forceinline__ unsigned pack2(float a, float b) {
  typedef _Float16 h2 __attribute__((ext_vector_type(2)));
  h2 v = {(_Float16)a, (_Float16)b};
  return __builtin_bit_cast(unsigned, v);
}

#define GLOAD2(D0, D1, BASE, VOFF, IMM) \
  asm volatile("global_load_dwordx4 %0, %2, %3 offset:%4 sc0 sc1\n\tglobal_load_dwordx4 %1, %2, %3 offset:%5 sc0 sc1" \
               : "=&v"(D0), "=&v"(D1) : "v"(VOFF), "s"(BASE), "i"(IMM), "i"(IMM + 1024) : "memory")
#define GSTORE2(D0, D1, BASE, VOFF, IMM) \
  asm volatile("global_store_dwordx4 %2, %0, %3 offset:%4 sc0 sc1\n\tglobal_store_dwordx4 %2, %1, %3 offset:%5 sc0 sc1\n\ts_nop 1" \
               :: "v"(D0), "v"(D1), "v"(VOFF), "s"(BASE), "i"(IMM), "i"(IMM + 1024) : "memory")
#define DRAW() do { if (lane == 0) asm volatile("global_atomic_add %0, %1, %2, %3 sc0" : "+v"(tv) : "v"(qoff), "v"(1u), "s"(lws) : "memory"); } while (0)
#define SIGNAL(OFFS) do { if (lane == 0) asm volatile("global_atomic_add %0, %1, %2" :: "v"(OFFS), "v"(1u), "s"(lws) : "memory"); } while (0)

#define STAMP(slot) do {} while (0)

template <int NI>
__device__ __forceinline__ void gemm_tile(f4 (&acc)[4][4], char* smem, char* lws, char* tmo, unsigned call, int nsh,
                                          const char* a_loc, const char* b_loc, int e_first, int xsel, int ct, int cs,
                                          int wave, int lane) {
  const int wm = wave >> 1, wn = wave & 1;
  #pragma unroll
  for (int mi = 0; mi < 4; ++mi)
    #pragma unroll
    for (int ni = 0; ni < NI; ++ni) acc[mi][ni] = f4{0.f, 0.f, 0.f, 0.f};
  const int nt = nsh * 8;
  const int aoff = wave * 4096 + lane * 16;
  const int boff = (NI == 4) ? wave * 2048 + lane * 16 : (wave >> 2) * 8192 + cs * 4096 + (wave & 3) * 1024 + lane * 16;
  const char* ap = a_loc + aoff;
  const char* bp = b_loc + boff;
#define ISSUE_STAGE(BUF) do { \
      char* dst_ = smem + (BUF) * STAGE_B; \
      _Pragma("unroll") for (int i = 0; i < 4; ++i) \
        __builtin_amdgcn_global_load_lds((const unsigned*)(ap + i * 1024), (unsigned*)(dst_ + wave * 4096 + i * 1024), 16, 0, 16); \
      _Pragma("unroll") for (int i = 0; i < NI / 2; ++i) \
        __builtin_amdgcn_global_load_lds((const unsigned*)(bp + i * 1024), (unsigned*)(dst_ + 32768 + wave * (512 * NI) + i * 1024), 16, 0, 16); \
      ap += 32768; bp += 16384; } while (0)
#define READ_FRAGS(AF, BF, BUF, KB) do { \
      const char* sa_ = smem + (BUF) * STAGE_B + (KB) * 16384 + wm * 4096 + lane * 16; \
      const char* sb_ = smem + (BUF) * STAGE_B + 32768 + (KB) * (2048 * NI) + wn * (1024 * NI) + lane * 16; \
      _Pragma("unroll") for (int i = 0; i < 4; ++i) { if (i < NI) BF[i] = *(const h8*)(sb_ + i * 1024); AF[i] = *(const h8*)(sa_ + i * 1024); } } while (0)
#define MMA(AF, BF) do { \
      __builtin_amdgcn_s_setprio(1); \
      _Pragma("unroll") for (int mi = 0; mi < 4; ++mi) \
        _Pragma("unroll") for (int ni = 0; ni < NI; ++ni) \
          acc[mi][ni] = __builtin_amdgcn_mfma_f32_16x16x32_f16(BF[ni], AF[mi], acc[mi][ni], 0, 0, 0); \
      __builtin_amdgcn_s_setprio(0); } while (0)
  h8 a0[4], b0[NI], a1[4], b1[NI];
  ISSUE_STAGE(0); ISSUE_STAGE(1); ISSUE_STAGE(2);
  if (NI == 4) asm volatile("s_waitcnt vmcnt(12)" ::: "memory");
  else asm volatile("s_waitcnt vmcnt(10)" ::: "memory");
  __builtin_amdgcn_s_barrier();
  READ_FRAGS(a0, b0, 0, 0);
  int buf = 0;
  for (int t = 0; t < nt; ++t) {
    READ_FRAGS(a1, b1, buf, 1);
    __builtin_amdgcn_sched_barrier(0);
    MMA(a0, b0);
    __builtin_amdgcn_sched_barrier(0);
    const int nbuf = (buf == 2) ? 0 : buf + 1;
    if (t + 1 < nt) {
      const int ti = t + 3;
      const bool next_shard = ti < nt && (ti & 7) == 0;
      int e = e_first - (ti >> 3); e += (e < 0) ? 7 : 0; e += 1;
      if (next_shard && wave == 0) {
        const char* word = (lane & 1) ? (lws + OFF_WCNT + (e * 16 + ct) * 128) : (lws + OFF_XCNT + (e * 16 + xsel) * 128);
        unsigned target = (lane & 1) ? call * 32u : call * 64u;
        (void)wave_wait<GSLEEP>(word, target, tmo, call);
      }
      if (t + 2 < nt) {
        if (NI == 4) asm volatile("s_waitcnt vmcnt(6) lgkmcnt(0)" ::: "memory");
        else asm volatile("s_waitcnt vmcnt(5) lgkmcnt(0)" ::: "memory");
      } else asm volatile("s_waitcnt vmcnt(0) lgkmcnt(0)" ::: "memory");
      __builtin_amdgcn_s_barrier();
      if (ti < nt) {
        if (next_shard) {
          ap = lws + OFF_XP + (size_t)(e * 16 + xsel) * 262144 + aoff;
          bp = lws + OFF_WP + (size_t)(e * 16 + ct) * 131072 + boff;
        }
        ISSUE_STAGE(buf);
      }
      READ_FRAGS(a0, b0, nbuf, 0);
    }
    __builtin_amdgcn_sched_barrier(0);
    MMA(a1, b1);
    __builtin_amdgcn_sched_barrier(0);
    buf = nbuf;
  }
}

__global__ void __launch_bounds__(NTHR, 2) fused_gemm_rs(Params P) {
  extern __shared__ __attribute__((aligned(16))) char smem[];
  const int tid = threadIdx.x, lane = tid & 63;
  const int wave = __builtin_amdgcn_readfirstlane(tid >> 6);
  const int bid = blockIdx.x;
  const int rank = P.rank;
  char* const lws = P.wsx[0];
  char* const tmo = lws + OFF_TMO;

  STAMP(0);
  const unsigned epoch = ld_agent(lws + OFF_EPOCH);
  f4 xv[2][2];
  {
    #pragma unroll
    for (int i = 0; i < 2; ++i) {
      int F = (bid * 8 + wave) * 2 + i;
      int rh = F >> 8, kb = (F >> 4) & 15, rb = F & 15;
      const float* src = P.x + (size_t)(rh * 256 + rb * 16 + (lane & 15)) * KS + kb * 32 + (lane >> 4) * 8;
      xv[i][0] = *(const f4*)src;
      xv[i][1] = *(const f4*)(src + 4);
    }
  }
  const unsigned call = __builtin_amdgcn_readfirstlane(epoch) + 1;
  const unsigned par = call & 1;

  auto Lr = __builtin_amdgcn_make_buffer_rsrc((void*)lws, 0, (int)WS_NEED, 0x00020000);

  {
    u4 o[2];
    #pragma unroll
    for (int i = 0; i < 2; ++i) {
      o[i][0] = pack2(xv[i][0][0], xv[i][0][1]);
      o[i][1] = pack2(xv[i][0][2], xv[i][0][3]);
      o[i][2] = pack2(xv[i][1][0], xv[i][1][1]);
      o[i][3] = pack2(xv[i][1][2], xv[i][1][3]);
    }
    const unsigned lofs = (unsigned)(OFF_XC + par * 4 * MiB) + (bid * 8 + wave) * 2048 + lane * 16;
    GSTORE2(o[0], o[1], lws, lofs, 0);
    VMCNT0();
    __builtin_amdgcn_s_barrier();
    if (wave == 0) SIGNAL(OFF_CX + (bid >> 4) * 128);
    if (bid == NWG - 1 && wave == 0) {
      const bool okx = wave_wait<1>(lws + OFF_CX + (lane & 15) * 128, call * 16u, tmo, call);
      if (okx && lane >= 1 && lane < 8) st_sys(peer_ws(P, lane) + OFF_XRDY + lane * 128, call);
      if (lane == 0) __hip_atomic_store((unsigned*)(lws + OFF_EPOCH), call, __ATOMIC_RELAXED, __HIP_MEMORY_SCOPE_AGENT);
      if (okx && wave_wait<4>(lws + OFF_CW + (lane & 7) * 128, call * (unsigned)(NGEMM / 8), tmo, call)) {
        if (lane < 8) st_sys((lane ? peer_ws(P, lane) : lws) + OFF_WRDY + lane * 128, call);
      }
    }
  }
  STAMP(1);

  int ue = 0;
  bool ok = true;
  unsigned tv = 0;
  const unsigned lo16 = lane * 16;
  if (bid < NGEMM) {
    {
      const int T0 = bid * 8 + wave;
      const int T2 = 2 * NGEMM * 8 + wave * NGEMM + bid;
      const bool third = T2 < 4096;
      float v[3][4][8];
      #pragma unroll
      for (int r = 0; r < 3; ++r) {
        if (r == 2 && !third) break;
        const int T = (r == 2) ? T2 : T0 + r * NGEMM * 8;
        const float* src = P.w + (size_t)((T >> 6) * 8) * NCOL + (T & 63) * 256 + lane;
        #pragma unroll
        for (int j = 0; j < 8; ++j)
          #pragma unroll
          for (int i = 0; i < 4; ++i) v[r][i][j] = src[(size_t)j * NCOL + i * 64];
      }
      #pragma unroll
      for (int r = 0; r < 3; ++r) {
        if (r == 2 && !third) break;
        const int T = (r == 2) ? T2 : T0 + r * NGEMM * 8;
        const int cg = T & 63, ko = T >> 6;
        int dst = (int)(OFF_WC + par * 16 * MiB) + cg * 2 * 131072 + (ko >> 2) * 8192 + (lane >> 4) * 1024 + ((ko & 3) * 16 + (lane & 15)) * 16;
        #pragma unroll
        for (int i = 0; i < 4; ++i) {
          u4 o;
          o[0] = pack2(v[r][i][0], v[r][i][1]);
          o[1] = pack2(v[r][i][2], v[r][i][3]);
          o[2] = pack2(v[r][i][4], v[r][i][5]);
          o[3] = pack2(v[r][i][6], v[r][i][7]);
          __builtin_amdgcn_raw_buffer_store_b128(o, Lr, dst + (i >> 1) * 131072 + (i & 1) * 4096, 0, WCAUX);
        }
      }
      VMCNT0();
      __syncthreads();
      if (wave == 0) SIGNAL(OFF_CW + (bid & 7) * 128);
    }
    STAMP(2);
    {
      if (wave == 0) (void)wave_wait<GSLEEP>(lws + OFF_WRDY, call, tmo, call);
      __syncthreads();
    }
    STAMP(3);

    const int xcd = bid & 7, slot = bid >> 3;
    const int ct = 4 * (xcd >> 1) + (slot & 3);
    const int h = xcd & 1, dq = (slot >> 2) + 1;
    const int wm = wave >> 1, wn = wave & 1;
    const char* const xc = lws + OFF_XC + par * 4 * MiB;
    const char* const wc = lws + OFF_WC + par * 16 * MiB;
    f4 acc[4][4];

    gemm_tile<4>(acc, smem, lws, tmo, call, 8, xc + (size_t)((rank ^ dq) * 2 + h) * 262144, wc + (size_t)(rank * 16 + ct) * 131072,
                 dq, dq * 2 + h, ct, 0, wave, lane);
    STAMP(4);
    {
      asm volatile("s_waitcnt lgkmcnt(0)" ::: "memory");
      __builtin_amdgcn_s_barrier();
      char* T = smem + (wm * 64 + (lane & 15)) * 528 + (wn * 64 + (lane >> 4) * 4) * 4;
      #pragma unroll
      for (int mi = 0; mi < 4; ++mi)
        #pragma unroll
        for (int ni = 0; ni < 4; ++ni) {
          f4 v = acc[mi][ni];
          #pragma unroll
          for (int k = 0; k < 4; ++k) v[k] = fmaxf(v[k], 0.f);
          *(f4*)(T + mi * 16 * 528 + ni * 64) = v;
        }
      __syncthreads();
      unsigned cnt0 = 0, cnt1 = 0;
      #pragma unroll 1
      for (int j = 0; j < 2; ++j) {
        char* R = smem + (wave * 32 + j * 16) * 528;
        unsigned off = 0, mlo = 0, mhi = 0;
        #pragma unroll 1
        for (int g0 = 0; g0 < 32; g0 += 8) {
          float v[8];
          #pragma unroll
          for (int i = 0; i < 8; ++i) v[i] = *(const float*)(R + ((g0 + i) >> 1) * 528 + ((i & 1) * 64 + lane) * 4);
          #pragma unroll
          for (int i = 0; i < 8; ++i) {
            const bool nz = v[i] != 0.f;
            const unsigned long long m = __ballot(nz);
            const unsigned pos = off + __builtin_amdgcn_mbcnt_hi((unsigned)(m >> 32), __builtin_amdgcn_mbcnt_lo((unsigned)m, 0u));
            if (nz) *(float*)(R + pos * 4) = v[i];
            off += __builtin_popcountll(m);
            if (lane == g0 + i) { mlo = (unsigned)m; mhi = (unsigned)(m >> 32); }
          }
        }
        const int slot_off = (int)(OFF_SP + par * SP_BYTES) + (((dq * 2 + h) * 16 + ct) * 16 + wave * 2 + j) * (int)CHUNK_B;
        if (lane < 32) {
          typedef unsigned u2 __attribute__((ext_vector_type(2)));
          __builtin_amdgcn_raw_buffer_store_b64(u2{mlo, mhi}, Lr, slot_off + lane * 8, 0, 17);
        }
        for (unsigned p = 0; p * 256 < off; ++p) {
          u4 d = *(const u4*)(R + p * 1024 + lane * 16);
          __builtin_amdgcn_raw_buffer_store_b128(d, Lr, slot_off + 256 + p * 1024 + lane * 16, 0, 17);
        }
        if (j == 0) cnt0 = off; else cnt1 = off;
      }
      VMCNT0();
      if (lane < 2 && ld_agent(tmo) != call) {
        unsigned long long gran = ((unsigned long long)(lane ? cnt1 : cnt0) << 32) | call;
        __hip_atomic_store((unsigned long long*)(peer_ws(P, dq) + OFF_FLC + (dq * 512 + (h * 16 + ct) * 16 + wave * 2 + lane) * 8), gran,
                           __ATOMIC_RELAXED, __HIP_MEMORY_SCOPE_SYSTEM);
      }
      __syncthreads();
    }
    STAMP(5);
    if (h == 0 && dq <= 4) {
      const int ho = (dq - 1) >> 1, cs = (dq - 1) & 1;
      gemm_tile<2>(acc, smem, lws, tmo, call, 8, xc + (size_t)(rank * 2 + ho) * 262144, wc + (size_t)(rank * 16 + ct) * 131072,
                   7, ho, ct, cs, wave, lane);
      const bool bad = ld_agent(tmo) == call;
      float* o = P.out + (size_t)(ho * 256 + wm * 64 + (lane & 15)) * NCOL + (rank * 16 + ct) * BN + cs * 64 + wn * 32 + (lane >> 4) * 4;
      #pragma unroll
      for (int mi = 0; mi < 4; ++mi)
        #pragma unroll
        for (int ni = 0; ni < 2; ++ni) {
          f4 v = acc[mi][ni];
          #pragma unroll
          for (int k = 0; k < 4; ++k) v[k] = bad ? __builtin_nanf("") : fmaxf(v[k], 0.f);
          OUT_STORE(o + (size_t)mi * 16 * NCOL + ni * 16, v);
        }
      STAMP(6);
    }
    if (h == 0 && dq >= 5) {
      const int hidx = (dq - 5) * 16 + ct;
      if (hidx < 7 * UHELP) {
        ue = hidx % 7 + 1;
        if (wave == 0) (void)wave_wait<GSLEEP>(lws + OFF_Q + ue * 128, (call - 1) * (unsigned)(U_C + PWAVES) + U_C, tmo, call);
        __syncthreads();
      }
    }
  } else if (bid < NGEMM + 28) {
    const int pw = bid - NGEMM;
    const int e = (pw % 7) + 1;
    char* const rws = peer_ws(P, e);
    const unsigned qbase = (call - 1) * (unsigned)(U_C + PWAVES);
    const unsigned qoff = OFF_Q + e * 128;
    ue = e;
    volatile unsigned* const gate = (volatile unsigned*)(smem + LDS_BC);
    if (tid == 0) *gate = 0;
    __syncthreads();
    if (wave < PW_PER_WG) {
    DRAW();
    ok = wave_wait<2>(lws + OFF_XRDY + e * 128, call, tmo, call);
    STAMP(2);
    asm volatile("s_waitcnt vmcnt(0)" : "+v"(tv) :: "memory");
    int t = (int)(__builtin_amdgcn_readfirstlane(tv) - qbase);
    u4 ra0, ra1, rb0, rb1;

#define CDESC(t, SO, LO, CN) do { \
      if ((t) >= U_W && (t) < U_X2) { \
        if (!wseen) { wseen = true; STAMP(3); ok = ok && wave_wait<2>(lws + OFF_WRDY + e * 128, call, tmo, call); } \
        int ct_ = ((t) - U_W) >> 5; \
        SO = (unsigned)(OFF_WC + par * 16 * MiB) + (rank * 16 + ct_) * 131072 + ((t) & 31) * 4096; \
        LO = (unsigned)OFF_WP + (e * 16 + ct_) * 131072 + ((t) & 31) * 4096; \
        CN = OFF_WCNT + (e * 16 + ct_) * 128; \
      } else { \
        const int hk_ = (int)(XSEQ >> (4 * (((t) < U_W) ? ((t) >> 6) : NXA + (((t) - U_X2) >> 6)))) & 15; \
        const int h_ = hk_ >> 3; \
        int dq_ = e - 1 + (hk_ & 7); dq_ -= (dq_ >= 7) ? 7 : 0; dq_ += 1; \
        if ((hk_ & 7) == 7) dq_ = 0; \
        SO = (unsigned)(OFF_XC + par * 4 * MiB) + ((rank ^ dq_) * 2 + h_) * 262144 + ((t) & 63) * 4096; \
        LO = (unsigned)OFF_XP + ((e * 8 + dq_) * 2 + h_) * 262144 + ((t) & 63) * 4096; \
        CN = OFF_XCNT + ((e * 8 + dq_) * 2 + h_) * 128; \
      } } while (0)
    if (ok && t < U_C) {
      bool wseen = false;
      unsigned pending = 0;
      unsigned soff, loff, cnt, nsoff = 0, nloff = 0, ncnt = 0;
      CDESC(t, soff, loff, cnt);
      GLOAD2(ra0, ra1, rws, soff + lo16, 0);
      DRAW();
      asm volatile("s_waitcnt vmcnt(0)" ::: "memory");
      for (;;) {
        GLOAD2(rb0, rb1, rws, soff + lo16, 2048);
        asm volatile("s_waitcnt vmcnt(4)" : "+v"(tv) :: "memory");
        GSTORE2(ra0, ra1, lws, loff + lo16, 0);
        t = (int)(__builtin_amdgcn_readfirstlane(tv) - qbase);
        const bool more = t < U_C;
        if (more) {
          CDESC(t, nsoff, nloff, ncnt);
          GLOAD2(ra0, ra1, rws, nsoff + lo16, 0);
          DRAW();
          asm volatile("s_waitcnt vmcnt(5)" ::: "memory");
        } else {
          asm volatile("s_waitcnt vmcnt(2)" ::: "memory");
        }
        if (pending) SIGNAL(pending);
        pending = cnt;
        GSTORE2(rb0, rb1, lws, loff + lo16, 2048);
        if (!more || !ok) break;
        soff = nsoff; loff = nloff; cnt = ncnt;
      }
      asm volatile("s_waitcnt vmcnt(0)" : "+v"(tv) :: "memory");
      if (ok) SIGNAL(pending);
      else {
        while (t < U_C) { t = (int)(__builtin_amdgcn_readfirstlane(tv) - qbase); if (t < U_C) { DRAW(); asm volatile("s_waitcnt vmcnt(0)" : "+v"(tv) :: "memory"); } }
      }
    } else if (!ok) {
      while (t < U_C) { DRAW(); asm volatile("s_waitcnt vmcnt(0)" : "+v"(tv) :: "memory"); t = (int)(__builtin_amdgcn_readfirstlane(tv) - qbase); }
    }
    if (lane == 0) *gate = 1;
    } else {
      while (*gate == 0) __builtin_amdgcn_s_sleep(8);
    }
    STAMP(4);
  }
  {
    if (ue != 0 && wave < UW_PER_WG) {
      const int e = ue;
      char* const rws = peer_ws(P, e);
      const unsigned qbase = (call - 1) * (unsigned)(NCHUNK + (4 + UHELP) * UW_PER_WG);
      const unsigned qoff = OFF_Q2 + e * 128;
      const unsigned bufa = wave * 17408;
      const char* const rsp = rws + OFF_SP + par * SP_BYTES + (size_t)e * NCHUNK * CHUNK_B;
      float* const outp = P.out + (rank ^ e) * 2048;
      int gbase = -1000;
      unsigned glo = 0, ghi = 0;
      unsigned lds_base;
      { typedef __attribute__((address_space(3))) char* lptr; lds_base = (unsigned)(size_t)(lptr)smem; }
#define FETCH(C, B) do { \
        unsigned cnt_ = 0; \
        unsigned long long t0_ = 0; bool timed_ = false; \
        for (;;) { \
          if ((C) >= gbase && (C) < gbase + 64 && (int)(__builtin_amdgcn_readlane(glo, (C) - gbase) - call) >= 0) { cnt_ = __builtin_amdgcn_readlane(ghi, (C) - gbase); break; } \
          if (timed_) { \
            if (ld_agent(tmo) == call || __builtin_amdgcn_s_memrealtime() - t0_ > DEADLINE_TICKS) { \
              __hip_atomic_store((unsigned*)tmo, call, __ATOMIC_RELAXED, __HIP_MEMORY_SCOPE_AGENT); \
              ok = false; break; \
            } \
            __builtin_amdgcn_s_sleep(8); \
          } else { timed_ = true; t0_ = __builtin_amdgcn_s_memrealtime(); } \
          gbase = (C); \
          int ci_ = (C) + lane; if (ci_ > NCHUNK - 1) ci_ = NCHUNK - 1; \
          unsigned long long gv_ = __hip_atomic_load((const unsigned long long*)(lws + OFF_FLC + (e * NCHUNK + ci_) * 8), __ATOMIC_RELAXED, __HIP_MEMORY_SCOPE_SYSTEM); \
          glo = (unsigned)gv_; ghi = (unsigned)(gv_ >> 32); \
        } \
        if (ok) { \
          const unsigned bytes_ = 256 + cnt_ * 4; \
          const char* src_ = rsp + (size_t)(C) * CHUNK_B + lo16; \
          for (unsigned p_ = 0; p_ < bytes_; p_ += 1024) { \
            if (p_ + lo16 < bytes_) { \
              unsigned keep_; \
              asm volatile("s_mov_b32 %0, m0\n\ts_mov_b32 m0, %2\n\ts_nop 0\n\tglobal_load_lds_dwordx4 %1, off sc0 sc1\n\ts_mov_b32 m0, %0" \
                           : "=&s"(keep_) : "v"(src_ + p_), "s"(lds_base + bufa + (B) * 8704 + p_) : "memory"); \
            } \
          } \
        } } while (0)
      DRAW();
      asm volatile("s_waitcnt vmcnt(0)" : "+v"(tv) :: "memory");
      int c = (int)(__builtin_amdgcn_readfirstlane(tv) - qbase);
      int cur = 0;
      if (c < NCHUNK) {
        if (ok) FETCH(c, 0);
        DRAW();
        asm volatile("s_waitcnt vmcnt(0)" ::: "memory");
      }
      while (c < NCHUNK) {
        asm volatile("s_waitcnt vmcnt(8)" : "+v"(tv) :: "memory");
        const bool good = ok;
        const int cn = (int)(__builtin_amdgcn_readfirstlane(tv) - qbase);
        if (cn < NCHUNK) {
          if (ok) FETCH(cn, cur ^ 1);
          DRAW();
        }
        float* o = outp + (size_t)((c >> 8) * 256 + (c & 15) * 16 + (lane >> 5)) * NCOL + ((c >> 4) & 15) * BN + (lane & 31) * 4;
        if (good) {
          const char* buf = smem + bufa + cur * 8704;
          typedef unsigned u2 __attribute__((ext_vector_type(2)));
          const u2 mk = *(const u2*)(buf + (lane & 31) * 8);
          unsigned pc = __builtin_popcount(mk[0]) + __builtin_popcount(mk[1]);
          unsigned inc = pc;
          #pragma unroll
          for (int d = 1; d < 32; d <<= 1) { unsigned y = __shfl_up(inc, d); if ((lane & 31) >= d) inc += y; }
          const unsigned base = inc - pc;
          const float* pk = (const float*)(buf + 256);
          const int tq = lane & 15;
          #pragma unroll
          for (int it = 0; it < 8; ++it) {
            const int g = it * 4 + (lane >> 4);
            const unsigned m0 = __shfl(mk[0], g), m1 = __shfl(mk[1], g), bs = __shfl(base, g);
            const unsigned long long m = ((unsigned long long)m1 << 32) | m0;
            const unsigned bits = (unsigned)(m >> (4 * tq)) & 15u;
            const unsigned p0 = bs + __builtin_popcountll(m & ((1ull << (4 * tq)) - 1ull));
            const unsigned p1 = p0 + (bits & 1u), p2 = p1 + ((bits >> 1) & 1u), p3 = p2 + ((bits >> 2) & 1u);
            f4 v;
            v[0] = pk[p0]; v[1] = pk[p1]; v[2] = pk[p2]; v[3] = pk[p3];
            v[0] = (bits & 1u) ? v[0] : 0.f; v[1] = (bits & 2u) ? v[1] : 0.f;
            v[2] = (bits & 4u) ? v[2] : 0.f; v[3] = (bits & 8u) ? v[3] : 0.f;
            OUT_STORE(o + (size_t)it * 2 * NCOL, v);
          }
        } else {
          float nv = __builtin_nanf("");
          f4 q = f4{nv, nv, nv, nv};
          #pragma unroll
          for (int it = 0; it < 8; ++it) OUT_STORE(o + (size_t)it * 2 * NCOL, q);
        }
        c = cn; cur ^= 1;
      }
    }
    if (bid >= NGEMM) STAMP(5);
  }
}

}

extern "C" void kernel_launch(int rank, int nranks, ncclComm_t comm,
                              hipStream_t stream,
                              void* const* d_in, const int* in_sizes, int n_in,
                              void* d_out, int out_size,
                              const int* in_dtypes, int out_dtype,
                              void* const* d_out_all,
                              void* const* d_ws_all, size_t ws_bytes) {
  (void)comm; (void)in_sizes; (void)n_in; (void)out_size; (void)in_dtypes; (void)out_dtype; (void)d_out_all;
  if (nranks != NRANK || ws_bytes < WS_NEED) {
    fprintf(stderr, "unsupported configuration: nranks %d ws %zu\n", nranks, ws_bytes);
    return;
  }
  HIP_CHECK(hipFuncSetAttribute((const void*)fused_gemm_rs, hipFuncAttributeMaxDynamicSharedMemorySize, LDS_BYTES));
  Params P;
  P.x = (const float*)d_in[0];
  P.w = (const float*)d_in[1];
  P.out = (float*)d_out;
  for (int e = 0; e < NRANK; ++e) P.wsx[e] = (char*)d_ws_all[rank ^ e];
  P.rank = rank;
  fused_gemm_rs<<<NWG, NTHR, LDS_BYTES, stream>>>(P);
  HIP_CHECK(hipGetLastError());
}
```

```cpp
#include <hip/hip_runtime.h>
#include <rccl/rccl.h>
#include <cstdio>
#include <cstdint>

#define HIP_CHECK(expr) do { hipError_t _e = (expr); if (_e != hipSuccess) { \
    fprintf(stderr, "HIP error %s at %s:%d\n", hipGetErrorString(_e), __FILE__, __LINE__); \
    return; } } while (0)

namespace {

typedef _Float16 h8 __attribute__((ext_vector_type(8)));
typedef float f4 __attribute__((ext_vector_type(4)));
typedef unsigned u4 __attribute__((ext_vector_type(4)));

constexpr int NRANK = 8;
constexpr int NCOL = 16384;
constexpr int KS = 512;
constexpr int BN = 128;
constexpr int NWG = 256;
constexpr int NGEMM = 224;
constexpr int NTHR = 512;
#define PW_PER_WG 5
constexpr int PWAVES = 4 * PW_PER_WG;

constexpr size_t MiB = 1024 * 1024;
constexpr unsigned OFF_EPOCH = 0;
constexpr unsigned OFF_TMO = 128;
constexpr unsigned OFF_XRDY = 1024;
constexpr unsigned OFF_WRDY = 2048;
constexpr unsigned OFF_Q = 3072;
constexpr unsigned OFF_CX = 4096;
constexpr unsigned OFF_CW = 6144;
constexpr unsigned OFF_XCNT = 8192;
constexpr unsigned OFF_WCNT = 32768;
constexpr unsigned OFF_Q2 = 7168;
constexpr unsigned OFF_FLC = 65536;
constexpr unsigned OFF_DBG = 131072;
constexpr size_t OFF_WC = 1 * MiB;
constexpr size_t OFF_XC = OFF_WC + 32 * MiB;
constexpr size_t OFF_WP = OFF_XC + 8 * MiB;
constexpr size_t OFF_XP = OFF_WP + 16 * MiB;
constexpr unsigned CHUNK_B = 8448;
constexpr size_t SP_BYTES = 34 * MiB;
constexpr size_t OFF_SP = OFF_XP + 32 * MiB;
constexpr size_t WS_NEED = OFF_SP + 2 * SP_BYTES;

constexpr unsigned STAGE_B = 49152;
constexpr unsigned LDS_BC = 3 * STAGE_B;
constexpr unsigned LDS_BYTES = LDS_BC + 64;

constexpr unsigned long long DEADLINE_TICKS = 50000000ull;

constexpr int NXA = 2, NXB = 14;
#define OWNPOS 12
constexpr unsigned long long XSEQ = OWNPOS == 14 ? 0xf7edcb65a4938210ull : OWNPOS == 12 ? 0xedf7cb65a4938210ull : OWNPOS == 11 ? 0xedcf7b65a4938210ull : 0xedcbf765a4938210ull;
constexpr int U_W = NXA * 64;
constexpr int U_X2 = U_W + 512;
constexpr int U_C = U_X2 + NXB * 64;
constexpr int NCHUNK = 512;

struct Params {
  const float* x;
  const float* w;
  float* out;
  char* wsx[NRANK];
  int rank;
};
__device__ __forceinline__ char* peer_ws(const Params& P, int e) {
  char* p = P.wsx[1];
  p = (e == 2) ? P.wsx[2] : p; p = (e == 3) ? P.wsx[3] : p; p = (e == 4) ? P.wsx[4] : p;
  p = (e == 5) ? P.wsx[5] : p; p = (e == 6) ? P.wsx[6] : p; p = (e == 7) ? P.wsx[7] : p;
  return p;
}

__device__ __forceinline__ unsigned ld_agent(const char* p) {
  return __hip_atomic_load((const unsigned*)p, __ATOMIC_RELAXED, __HIP_MEMORY_SCOPE_AGENT);
}
__device__ __forceinline__ unsigned ld_sys(const char* p) {
  return __hip_atomic_load((const unsigned*)p, __ATOMIC_RELAXED, __HIP_MEMORY_SCOPE_SYSTEM);
}
__device__ __forceinline__ void st_sys(char* p, unsigned v) {
  __hip_atomic_store((unsigned*)p, v, __ATOMIC_RELAXED, __HIP_MEMORY_SCOPE_SYSTEM);
}
#define VMCNT0() asm volatile("s_waitcnt vmcnt(0)" ::: "memory")

template <int SLEEP>
__device__ __forceinline__ bool wave_wait(const char* word, unsigned target, char* tmo, unsigned call) {
  unsigned v = ld_sys(word);
  if (__all((int)(v - target) >= 0)) return true;
  unsigned long long t0 = __builtin_amdgcn_s_memrealtime();
  for (;;) {
    __builtin_amdgcn_s_sleep(SLEEP);
    v = ld_sys(word);
    if (__all((int)(v - target) >= 0)) return true;
    if (ld_agent(tmo) == call || __builtin_amdgcn_s_memrealtime() - t0 > DEADLINE_TICKS) {
      __hip_atomic_store((unsigned*)tmo, call, __ATOMIC_RELAXED, __HIP_MEMORY_SCOPE_AGENT);
      return false;
    }
  }
}
#define UW_PER_WG 3
#define UHELP 4
#define OUTMODE 2
#define OUT_STORE(P_, V_) asm volatile("global_store_dwordx4 %0, %1, off sc0 sc1\n\ts_nop 1" :: "v"(P_), "v"(V_) : "memory")
#define GSLEEP 32

__device__ __forceinline__ unsigned pack2(float a, float b) {
  typedef _Float16 h2 __attribute__((ext_vector_type(2)));
  h2 v = {(_Float16)a, (_Float16)b};
  return __builtin_bit_cast(unsigned, v);
}

#define GLOAD2(D0, D1, BASE, VOFF, IMM) \
  asm volatile("global_load_dwordx4 %0, %2, %3 offset:%4 sc0 sc1\n\tglobal_load_dwordx4 %1, %2, %3 offset:%5 sc0 sc1" \
               : "=&v"(D0), "=&v"(D1) : "v"(VOFF), "s"(BASE), "i"(IMM), "i"(IMM + 1024) : "memory")
#define GSTORE2(D0, D1, BASE, VOFF, IMM) \
  asm volatile("global_store_dwordx4 %2, %0, %3 offset:%4 sc0 sc1\n\tglobal_store_dwordx4 %2, %1, %3 offset:%5 sc0 sc1\n\ts_nop 1" \
               :: "v"(D0), "v"(D1), "v"(VOFF), "s"(BASE), "i"(IMM), "i"(IMM + 1024) : "memory")
#define DRAW() do { if (lane == 0) asm volatile("global_atomic_add %0, %1, %2, %3 sc0" : "+v"(tv) : "v"(qoff), "v"(1u), "s"(lws) : "memory"); } while (0)
#define SIGNAL(OFFS) do { if (lane == 0) asm volatile("global_atomic_add %0, %1, %2" :: "v"(OFFS), "v"(1u), "s"(lws) : "memory"); } while (0)

#define STAMP(slot) do {} while (0)

template <int NI>
__device__ __forceinline__ void gemm_tile(f4 (&acc)[4][4], char* smem, char* lws, char* tmo, unsigned call, int nsh,
                                          const char* a_loc, const char* b_loc, int e_first, int xsel, int ct, int cs,
                                          int wave, int lane) {
  const int wm = wave >> 1, wn = wave & 1;
  #pragma unroll
  for (int mi = 0; mi < 4; ++mi)
    #pragma unroll
    for (int ni = 0; ni < NI; ++ni) acc[mi][ni] = f4{0.f, 0.f, 0.f, 0.f};
  const int nt = nsh * 8;
  const int aoff = wave * 4096 + lane * 16;
  const int boff = (NI == 4) ? wave * 2048 + lane * 16 : (wave >> 2) * 8192 + cs * 4096 + (wave & 3) * 1024 + lane * 16;
  const char* ap = a_loc + aoff;
  const char* bp = b_loc + boff;
#define ISSUE_STAGE(BUF) do { \
      char* dst_ = smem + (BUF) * STAGE_B; \
      _Pragma("unroll") for (int i = 0; i < 4; ++i) \
        __builtin_amdgcn_global_load_lds((const unsigned*)(ap + i * 1024), (unsigned*)(dst_ + wave * 4096 + i * 1024), 16, 0, 16); \
      _Pragma("unroll") for (int i = 0; i < NI / 2; ++i) \
        __builtin_amdgcn_global_load_lds((const unsigned*)(bp + i * 1024), (unsigned*)(dst_ + 32768 + wave * (512 * NI) + i * 1024), 16, 0, 16); \
      ap += 32768; bp += 16384; } while (0)
#define READ_FRAGS(AF, BF, BUF, KB) do { \
      const char* sa_ = smem + (BUF) * STAGE_B + (KB) * 16384 + wm * 4096 + lane * 16; \
      const char* sb_ = smem + (BUF) * STAGE_B + 32768 + (KB) * (2048 * NI) + wn * (1024 * NI) + lane * 16; \
      _Pragma("unroll") for (int i = 0; i < 4; ++i) { if (i < NI) BF[i] = *(const h8*)(sb_ + i * 1024); AF[i] = *(const h8*)(sa_ + i * 1024); } } while (0)
#define MMA(AF, BF) do { \
      __builtin_amdgcn_s_setprio(1); \
      _Pragma("unroll") for (int mi = 0; mi < 4; ++mi) \
        _Pragma("unroll") for (int ni = 0; ni < NI; ++ni) \
          acc[mi][ni] = __builtin_amdgcn_mfma_f32_16x16x32_f16(BF[ni], AF[mi], acc[mi][ni], 0, 0, 0); \
      __builtin_amdgcn_s_setprio(0); } while (0)
  h8 a0[4], b0[NI], a1[4], b1[NI];
  ISSUE_STAGE(0); ISSUE_STAGE(1); ISSUE_STAGE(2);
  if (NI == 4) asm volatile("s_waitcnt vmcnt(12)" ::: "memory");
  else asm volatile("s_waitcnt vmcnt(10)" ::: "memory");
  __builtin_amdgcn_s_barrier();
  READ_FRAGS(a0, b0, 0, 0);
  int buf = 0;
  for (int t = 0; t < nt; ++t) {
    READ_FRAGS(a1, b1, buf, 1);
    __builtin_amdgcn_sched_barrier(0);
    MMA(a0, b0);
    __builtin_amdgcn_sched_barrier(0);
    const int nbuf = (buf == 2) ? 0 : buf + 1;
    if (t + 1 < nt) {
      const int ti = t + 3;
      const bool next_shard = ti < nt && (ti & 7) == 0;
      int e = e_first - (ti >> 3); e += (e < 0) ? 7 : 0; e += 1;
      if (next_shard && wave == 0) {
        const char* word = (lane & 1) ? (lws + OFF_WCNT + (e * 16 + ct) * 128) : (lws + OFF_XCNT + (e * 16 + xsel) * 128);
        unsigned target = (lane & 1) ? call * 32u : call * 64u;
        (void)wave_wait<GSLEEP>(word, target, tmo, call);
      }
      if (t + 2 < nt) {
        if (NI == 4) asm volatile("s_waitcnt vmcnt(6) lgkmcnt(0)" ::: "memory");
        else asm volatile("s_waitcnt vmcnt(5) lgkmcnt(0)" ::: "memory");
      } else asm volatile("s_waitcnt vmcnt(0) lgkmcnt(0)" ::: "memory");
      __builtin_amdgcn_s_barrier();
      if (ti < nt) {
        if (next_shard) {
          ap = lws + OFF_XP + (size_t)(e * 16 + xsel) * 262144 + aoff;
          bp = lws + OFF_WP + (size_t)(e * 16 + ct) * 131072 + boff;
        }
        ISSUE_STAGE(buf);
      }
      READ_FRAGS(a0, b0, nbuf, 0);
    }
    __builtin_amdgcn_sched_barrier(0);
    MMA(a1, b1);
    __builtin_amdgcn_sched_barrier(0);
    buf = nbuf;
  }
}

__global__ void __launch_bounds__(NTHR, 2) fused_gemm_rs(Params P) {
  extern __shared__ __attribute__((aligned(16))) char smem[];
  const int tid = threadIdx.x, lane = tid & 63;
  const int wave = __builtin_amdgcn_readfirstlane(tid >> 6);
  const int bid = blockIdx.x;
  const int rank = P.rank;
  char* const lws = P.wsx[0];
  char* const tmo = lws + OFF_TMO;

  STAMP(0);
  const unsigned epoch = ld_agent(lws + OFF_EPOCH);
  f4 xv[2][2];
  {
    #pragma unroll
    for (int i = 0; i < 2; ++i) {
      int F = (bid * 8 + wave) * 2 + i;
      int rh = F >> 8, kb = (F >> 4) & 15, rb = F & 15;
      const float* src = P.x + (size_t)(rh * 256 + rb * 16 + (lane & 15)) * KS + kb * 32 + (lane >> 4) * 8;
      xv[i][0] = *(const f4*)src;
      xv[i][1] = *(const f4*)(src + 4);
    }
  }
  const unsigned call = __builtin_amdgcn_readfirstlane(epoch) + 1;
  const unsigned par = call & 1;

  auto Lr = __builtin_amdgcn_make_buffer_rsrc((void*)lws, 0, (int)WS_NEED, 0x00020000);

  {
    u4 o[2];
    #pragma unroll
    for (int i = 0; i < 2; ++i) {
      o[i][0] = pack2(xv[i][0][0], xv[i][0][1]);
      o[i][1] = pack2(xv[i][0][2], xv[i][0][3]);
      o[i][2] = pack2(xv[i][1][0], xv[i][1][1]);
      o[i][3] = pack2(xv[i][1][2], xv[i][1][3]);
    }
    const unsigned lofs = (unsigned)(OFF_XC + par * 4 * MiB) + (bid * 8 + wave) * 2048 + lane * 16;
    GSTORE2(o[0], o[1], lws, lofs, 0);
    VMCNT0();
    __builtin_amdgcn_s_barrier();
    if (wave == 0) SIGNAL(OFF_CX + (bid >> 4) * 128);
    if (bid == NWG - 1 && wave == 0) {
      const bool okx = wave_wait<1>(lws + OFF_CX + (lane & 15) * 128, call * 16u, tmo, call);
      if (okx && lane >= 1 && lane < 8) st_sys(peer_ws(P, lane) + OFF_XRDY + lane * 128, call);
      if (lane == 0) __hip_atomic_store((unsigned*)(lws + OFF_EPOCH), call, __ATOMIC_RELAXED, __HIP_MEMORY_SCOPE_AGENT);
      if (okx && wave_wait<4>(lws + OFF_CW + (lane & 7) * 128, call * (unsigned)(NGEMM / 8), tmo, call)) {
        if (lane < 8) st_sys((lane ? peer_ws(P, lane) : lws) + OFF_WRDY + lane * 128, call);
      }
    }
  }
  STAMP(1);

  int ue = 0;
  bool ok = true;
  unsigned tv = 0;
  const unsigned lo16 = lane * 16;
  if (bid < NGEMM) {
    {
      const int T0 = bid * 8 + wave;
      const int T2 = 2 * NGEMM * 8 + wave * NGEMM + bid;
      const bool third = T2 < 4096;
      float v[3][4][8];
      #pragma unroll
      for (int r = 0; r < 3; ++r) {
        if (r == 2 && !third) break;
        const int T = (r == 2) ? T2 : T0 + r * NGEMM * 8;
        const float* src = P.w + (size_t)((T >> 6) * 8) * NCOL + (T & 63) * 256 + lane;
        #pragma unroll
        for (int j = 0; j < 8; ++j)
          #pragma unroll
          for (int i = 0; i < 4; ++i) v[r][i][j] = src[(size_t)j * NCOL + i * 64];
      }
      #pragma unroll
      for (int r = 0; r < 3; ++r) {
        if (r == 2 && !third) break;
        const int T = (r == 2) ? T2 : T0 + r * NGEMM * 8;
        const int cg = T & 63, ko = T >> 6;
        int dst = (int)(OFF_WC + par * 16 * MiB) + cg * 2 * 131072 + (ko >> 2) * 8192 + (lane >> 4) * 1024 + ((ko & 3) * 16 + (lane & 15)) * 16;
        #pragma unroll
        for (int i = 0; i < 4; ++i) {
          u4 o;
          o[0] = pack2(v[r][i][0], v[r][i][1]);
          o[1] = pack2(v[r][i][2], v[r][i][3]);
          o[2] = pack2(v[r][i][4], v[r][i][5]);
          o[3] = pack2(v[r][i][6], v[r][i][7]);
          __builtin_amdgcn_raw_buffer_store_b128(o, Lr, dst + (i >> 1) * 131072 + (i & 1) * 4096, 0, 17);
        }
      }
      VMCNT0();
      __syncthreads();
      if (wave == 0) SIGNAL(OFF_CW + (bid & 7) * 128);
    }
    STAMP(2);
    {
      if (wave == 0) (void)wave_wait<GSLEEP>(lws + OFF_WRDY, call, tmo, call);
      __syncthreads();
    }
    STAMP(3);

    const int xcd = bid & 7, slot = bid >> 3;
    const int ct = 4 * (xcd >> 1) + (slot & 3);
    const int h = xcd & 1, dq = (slot >> 2) + 1;
    const int wm = wave >> 1, wn = wave & 1;
    const char* const xc = lws + OFF_XC + par * 4 * MiB;
    const char* const wc = lws + OFF_WC + par * 16 * MiB;
    f4 acc[4][4];

    gemm_tile<4>(acc, smem, lws, tmo, call, 8, xc + (size_t)((rank ^ dq) * 2 + h) * 262144, wc + (size_t)(rank * 16 + ct) * 131072,
                 dq, dq * 2 + h, ct, 0, wave, lane);
    STAMP(4);
    {
      asm volatile("s_waitcnt lgkmcnt(0)" ::: "memory");
      __builtin_amdgcn_s_barrier();
      char* T = smem + (wm * 64 + (lane & 15)) * 528 + (wn * 64 + (lane >> 4) * 4) * 4;
      #pragma unroll
      for (int mi = 0; mi < 4; ++mi)
        #pragma unroll
        for (int ni = 0; ni < 4; ++ni) {
          f4 v = acc[mi][ni];
          #pragma unroll
          for (int k = 0; k < 4; ++k) v[k] = fmaxf(v[k], 0.f);
          *(f4*)(T + mi * 16 * 528 + ni * 64) = v;
        }
      __syncthreads();
      unsigned cnt0 = 0, cnt1 = 0;
      #pragma unroll 1
      for (int j = 0; j < 2; ++j) {
        char* R = smem + (wave * 32 + j * 16) * 528;
        unsigned off = 0, mlo = 0, mhi = 0;
        #pragma unroll 1
        for (int g0 = 0; g0 < 32; g0 += 8) {
          float v[8];
          #pragma unroll
          for (int i = 0; i < 8; ++i) v[i] = *(const float*)(R + ((g0 + i) >> 1) * 528 + ((i & 1) * 64 + lane) * 4);
          #pragma unroll
          for (int i = 0; i < 8; ++i) {
            const bool nz = v[i] != 0.f;
            const unsigned long long m = __ballot(nz);
            const unsigned pos = off + __builtin_amdgcn_mbcnt_hi((unsigned)(m >> 32), __builtin_amdgcn_mbcnt_lo((unsigned)m, 0u));
            if (nz) *(float*)(R + pos * 4) = v[i];
            off += __builtin_popcountll(m);
            if (lane == g0 + i) { mlo = (unsigned)m; mhi = (unsigned)(m >> 32); }
          }
        }
        const int slot_off = (int)(OFF_SP + par * SP_BYTES) + (((dq * 2 + h) * 16 + ct) * 16 + wave * 2 + j) * (int)CHUNK_B;
        if (lane < 32) {
          typedef unsigned u2 __attribute__((ext_vector_type(2)));
          __builtin_amdgcn_raw_buffer_store_b64(u2{mlo, mhi}, Lr, slot_off + lane * 8, 0, 17);
        }
        for (unsigned p = 0; p * 256 < off; ++p) {
          u4 d = *(const u4*)(R + p * 1024 + lane * 16);
          __builtin_amdgcn_raw_buffer_store_b128(d, Lr, slot_off + 256 + p * 1024 + lane * 16, 0, 17);
        }
        if (j == 0) cnt0 = off; else cnt1 = off;
      }
      VMCNT0();
      if (lane < 2 && ld_agent(tmo) != call) {
        unsigned long long gran = ((unsigned long long)(lane ? cnt1 : cnt0) << 32) | call;
        __hip_atomic_store((unsigned long long*)(peer_ws(P, dq) + OFF_FLC + (dq * 512 + (h * 16 + ct) * 16 + wave * 2 + lane) * 8), gran,
                           __ATOMIC_RELAXED, __HIP_MEMORY_SCOPE_SYSTEM);
      }
      __syncthreads();
    }
    STAMP(5);
    if (h == 0 && dq <= 4) {
      const int ho = (dq - 1) >> 1, cs = (dq - 1) & 1;
      gemm_tile<2>(acc, smem, lws, tmo, call, 8, xc + (size_t)(rank * 2 + ho) * 262144, wc + (size_t)(rank * 16 + ct) * 131072,
                   7, ho, ct, cs, wave, lane);
      const bool bad = ld_agent(tmo) == call;
      float* o = P.out + (size_t)(ho * 256 + wm * 64 + (lane & 15)) * NCOL + (rank * 16 + ct) * BN + cs * 64 + wn * 32 + (lane >> 4) * 4;
      #pragma unroll
      for (int mi = 0; mi < 4; ++mi)
        #pragma unroll
        for (int ni = 0; ni < 2; ++ni) {
          f4 v = acc[mi][ni];
          #pragma unroll
          for (int k = 0; k < 4; ++k) v[k] = bad ? __builtin_nanf("") : fmaxf(v[k], 0.f);
          OUT_STORE(o + (size_t)mi * 16 * NCOL + ni * 16, v);
        }
      STAMP(6);
    }
    if (h == 0 && dq >= 5) {
      const int hidx = (dq - 5) * 16 + ct;
      if (hidx < 7 * UHELP) {
        ue = hidx % 7 + 1;
        if (wave == 0) (void)wave_wait<GSLEEP>(lws + OFF_Q + ue * 128, (call - 1) * (unsigned)(U_C + PWAVES) + U_C, tmo, call);
        __syncthreads();
      }
    }
  } else if (bid < NGEMM + 28) {
    const int pw = bid - NGEMM;
    const int e = (pw % 7) + 1;
    char* const rws = peer_ws(P, e);
    const unsigned qbase = (call - 1) * (unsigned)(U_C + PWAVES);
    const unsigned qoff = OFF_Q + e * 128;
    ue = e;
    volatile unsigned* const gate = (volatile unsigned*)(smem + LDS_BC);
    if (tid == 0) *gate = 0;
    __syncthreads();
    if (wave < PW_PER_WG) {
    DRAW();
    ok = wave_wait<2>(lws + OFF_XRDY + e * 128, call, tmo, call);
    STAMP(2);
    asm volatile("s_waitcnt vmcnt(0)" : "+v"(tv) :: "memory");
    int t = (int)(__builtin_amdgcn_readfirstlane(tv) - qbase);
    u4 ra0, ra1, rb0, rb1;

#define CDESC(t, SO, LO, CN) do { \
      if ((t) >= U_W && (t) < U_X2) { \
        if (!wseen) { wseen = true; STAMP(3); ok = ok && wave_wait<2>(lws + OFF_WRDY + e * 128, call, tmo, call); } \
        int ct_ = ((t) - U_W) >> 5; \
        SO = (unsigned)(OFF_WC + par * 16 * MiB) + (rank * 16 + ct_) * 131072 + ((t) & 31) * 4096; \
        LO = (unsigned)OFF_WP + (e * 16 + ct_) * 131072 + ((t) & 31) * 4096; \
        CN = OFF_WCNT + (e * 16 + ct_) * 128; \
      } else { \
        const int hk_ = (int)(XSEQ >> (4 * (((t) < U_W) ? ((t) >> 6) : NXA + (((t) - U_X2) >> 6)))) & 15; \
        const int h_ = hk_ >> 3; \
        int dq_ = e - 1 + (hk_ & 7); dq_ -= (dq_ >= 7) ? 7 : 0; dq_ += 1; \
        if ((hk_ & 7) == 7) dq_ = 0; \
        SO = (unsigned)(OFF_XC + par * 4 * MiB) + ((rank ^ dq_) * 2 + h_) * 262144 + ((t) & 63) * 4096; \
        LO = (unsigned)OFF_XP + ((e * 8 + dq_) * 2 + h_) * 262144 + ((t) & 63) * 4096; \
        CN = OFF_XCNT + ((e * 8 + dq_) * 2 + h_) * 128; \
      } } while (0)
    if (ok && t < U_C) {
      bool wseen = false;
      unsigned pending = 0;
      unsigned soff, loff, cnt, nsoff = 0, nloff = 0, ncnt = 0;
      CDESC(t, soff, loff, cnt);
      GLOAD2(ra0, ra1, rws, soff + lo16, 0);
      DRAW();
      asm volatile("s_waitcnt vmcnt(0)" ::: "memory");
      for (;;) {
        GLOAD2(rb0, rb1, rws, soff + lo16, 2048);
        asm volatile("s_waitcnt vmcnt(4)" : "+v"(tv) :: "memory");
        GSTORE2(ra0, ra1, lws, loff + lo16, 0);
        t = (int)(__builtin_amdgcn_readfirstlane(tv) - qbase);
        const bool more = t < U_C;
        if (more) {
          CDESC(t, nsoff, nloff, ncnt);
          GLOAD2(ra0, ra1, rws, nsoff + lo16, 0);
          DRAW();
          asm volatile("s_waitcnt vmcnt(5)" ::: "memory");
        } else {
          asm volatile("s_waitcnt vmcnt(2)" ::: "memory");
        }
        if (pending) SIGNAL(pending);
        pending = cnt;
        GSTORE2(rb0, rb1, lws, loff + lo16, 2048);
        if (!more || !ok) break;
        soff = nsoff; loff = nloff; cnt = ncnt;
      }
      asm volatile("s_waitcnt vmcnt(0)" : "+v"(tv) :: "memory");
      if (ok) SIGNAL(pending);
      else {
        while (t < U_C) { t = (int)(__builtin_amdgcn_readfirstlane(tv) - qbase); if (t < U_C) { DRAW(); asm volatile("s_waitcnt vmcnt(0)" : "+v"(tv) :: "memory"); } }
      }
    } else if (!ok) {
      while (t < U_C) { DRAW(); asm volatile("s_waitcnt vmcnt(0)" : "+v"(tv) :: "memory"); t = (int)(__builtin_amdgcn_readfirstlane(tv) - qbase); }
    }
    if (lane == 0) *gate = 1;
    } else if (wave < UW_PER_WG) {
      while (*gate == 0) __builtin_amdgcn_s_sleep(8);
    }
    STAMP(4);
  }
  {
    if (ue != 0 && wave < UW_PER_WG) {
      const int e = ue;
      char* const rws = peer_ws(P, e);
      const unsigned qbase = (call - 1) * (unsigned)(NCHUNK + (4 + UHELP) * UW_PER_WG);
      const unsigned qoff = OFF_Q2 + e * 128;
      const unsigned bufa = wave * 17408;
      const char* const rsp = rws + OFF_SP + par * SP_BYTES + (size_t)e * NCHUNK * CHUNK_B;
      float* const outp = P.out + (rank ^ e) * 2048;
      int gbase = -1000;
      unsigned glo = 0, ghi = 0;
      unsigned lds_base;
      { typedef __attribute__((address_space(3))) char* lptr; lds_base = (unsigned)(size_t)(lptr)smem; }
#define FETCH(C, B) do { \
        unsigned cnt_ = 0; \
        unsigned long long t0_ = 0; bool timed_ = false; \
        for (;;) { \
          if ((C) >= gbase && (C) < gbase + 64 && (int)(__builtin_amdgcn_readlane(glo, (C) - gbase) - call) >= 0) { cnt_ = __builtin_amdgcn_readlane(ghi, (C) - gbase); break; } \
          if (timed_) { \
            if (ld_agent(tmo) == call || __builtin_amdgcn_s_memrealtime() - t0_ > DEADLINE_TICKS) { \
              __hip_atomic_store((unsigned*)tmo, call, __ATOMIC_RELAXED, __HIP_MEMORY_SCOPE_AGENT); \
              ok = false; break; \
            } \
            __builtin_amdgcn_s_sleep(8); \
          } else { timed_ = true; t0_ = __builtin_amdgcn_s_memrealtime(); } \
          gbase = (C); \
          int ci_ = (C) + lane; if (ci_ > NCHUNK - 1) ci_ = NCHUNK - 1; \
          unsigned long long gv_ = __hip_atomic_load((const unsigned long long*)(lws + OFF_FLC + (e * NCHUNK + ci_) * 8), __ATOMIC_RELAXED, __HIP_MEMORY_SCOPE_SYSTEM); \
          glo = (unsigned)gv_; ghi = (unsigned)(gv_ >> 32); \
        } \
        if (ok) { \
          const unsigned bytes_ = 256 + (cnt_ < 2048u ? cnt_ : 2048u) * 4; \
          const char* src_ = rsp + (size_t)(C) * CHUNK_B + lo16; \
          for (unsigned p_ = 0; p_ < bytes_; p_ += 1024) { \
            if (p_ + lo16 < bytes_) { \
              unsigned keep_; \
              asm volatile("s_mov_b32 %0, m0\n\ts_mov_b32 m0, %2\n\ts_nop 0\n\tglobal_load_lds_dwordx4 %1, off sc0 sc1\n\ts_mov_b32 m0, %0" \
                           : "=&s"(keep_) : "v"(src_ + p_), "s"(lds_base + bufa + (B) * 8704 + p_) : "memory"); \
            } \
          } \
        } } while (0)
      DRAW();
      asm volatile("s_waitcnt vmcnt(0)" : "+v"(tv) :: "memory");
      int c = (int)(__builtin_amdgcn_readfirstlane(tv) - qbase);
      int cur = 0;
      if (c < NCHUNK) {
        if (ok) FETCH(c, 0);
        DRAW();
        asm volatile("s_waitcnt vmcnt(0)" ::: "memory");
      }
      while (c < NCHUNK) {
        asm volatile("s_waitcnt vmcnt(8)" : "+v"(tv) :: "memory");
        const bool good = ok;
        const int cn = (int)(__builtin_amdgcn_readfirstlane(tv) - qbase);
        if (cn < NCHUNK) {
          if (ok) FETCH(cn, cur ^ 1);
          DRAW();
        }
        float* o = outp + (size_t)((c >> 8) * 256 + (c & 15) * 16 + (lane >> 5)) * NCOL + ((c >> 4) & 15) * BN + (lane & 31) * 4;
        if (good) {
          const char* buf = smem + bufa + cur * 8704;
          typedef unsigned u2 __attribute__((ext_vector_type(2)));
          const u2 mk = *(const u2*)(buf + (lane & 31) * 8);
          unsigned pc = __builtin_popcount(mk[0]) + __builtin_popcount(mk[1]);
          unsigned inc = pc;
          #pragma unroll
          for (int d = 1; d < 32; d <<= 1) { unsigned y = __shfl_up(inc, d); if ((lane & 31) >= d) inc += y; }
          const unsigned base = inc - pc;
          const float* pk = (const float*)(buf + 256);
          const int tq = lane & 15;
          #pragma unroll
          for (int it = 0; it < 8; ++it) {
            const int g = it * 4 + (lane >> 4);
            const unsigned m0 = __shfl(mk[0], g), m1 = __shfl(mk[1], g), bs = __shfl(base, g);
            const unsigned long long m = ((unsigned long long)m1 << 32) | m0;
            const unsigned bits = (unsigned)(m >> (4 * tq)) & 15u;
            const unsigned p0 = bs + __builtin_popcountll(m & ((1ull << (4 * tq)) - 1ull));
            const unsigned p1 = p0 + (bits & 1u), p2 = p1 + ((bits >> 1) & 1u), p3 = p2 + ((bits >> 2) & 1u);
            f4 v;
            v[0] = pk[p0]; v[1] = pk[p1]; v[2] = pk[p2]; v[3] = pk[p3];
            v[0] = (bits & 1u) ? v[0] : 0.f; v[1] = (bits & 2u) ? v[1] : 0.f;
            v[2] = (bits & 4u) ? v[2] : 0.f; v[3] = (bits & 8u) ? v[3] : 0.f;
            OUT_STORE(o + (size_t)it * 2 * NCOL, v);
          }
        } else {
          float nv = __builtin_nanf("");
          f4 q = f4{nv, nv, nv, nv};
          #pragma unroll
          for (int it = 0; it < 8; ++it) OUT_STORE(o + (size_t)it * 2 * NCOL, q);
        }
        c = cn; cur ^= 1;
      }
    }
    if (bid >= NGEMM) STAMP(5);
  }
}

}

extern "C" void kernel_launch(int rank, int nranks, ncclComm_t comm,
                              hipStream_t stream,
                              void* const* d_in, const int* in_sizes, int n_in,
                              void* d_out, int out_size,
                              const int* in_dtypes, int out_dtype,
                              void* const* d_out_all,
                              void* const* d_ws_all, size_t ws_bytes) {
  (void)comm; (void)in_sizes; (void)n_in; (void)out_size; (void)in_dtypes; (void)out_dtype; (void)d_out_all;
  if (nranks != NRANK || ws_bytes < WS_NEED) {
    fprintf(stderr, "unsupported configuration: nranks %d ws %zu\n", nranks, ws_bytes);
    return;
  }
  HIP_CHECK(hipFuncSetAttribute((const void*)fused_gemm_rs, hipFuncAttributeMaxDynamicSharedMemorySize, LDS_BYTES));
  Params P;
  P.x = (const float*)d_in[0];
  P.w = (const float*)d_in[1];
  P.out = (float*)d_out;
  for (int e = 0; e < NRANK; ++e) P.wsx[e] = (char*)d_ws_all[rank ^ e];
  P.rank = rank;
  fused_gemm_rs<<<NWG, NTHR, LDS_BYTES, stream>>>(P);
  HIP_CHECK(hipGetLastError());
}
```
